```python
import numpy as np

kernel_name = "gpu_a2a_gemm_m4096_k4096_n4096_e4m3e5m2_dqsilu"


ABSMAX_RTOL = 0.05

COMPUTE_DTYPE = "float32"
EPILOGUE = "dequant_silu"
IN_QUANT = None
OUT_QUANT = None
IN_FP8 = "e4m3e5m2"
IN_DTYPES = {"x": "e4m3", "w_mat": "e5m2",
             "scale_x": "float32", "scale_w": "float32"}
WORLD_SIZE = 8

def _snap_fp8(x, mant_bits, max_abs, min_normal):
    a = np.abs(x.astype(np.float64))
    _m, _e = np.frexp(a)
    q = np.where(a >= min_normal, np.ldexp(1.0, _e - 1 - mant_bits),
                 min_normal * 2.0 ** -mant_bits)
    snapped = np.minimum(np.round(a / q) * q, max_abs)
    return (np.sign(x) * snapped).astype(np.float32)

def setup_inputs():

    x = _snap_fp8(np.random.randn(4096, 4096).astype(np.float32), 3, 448.0, 2.0 ** -6)
    w_mat = _snap_fp8(np.random.randn(4096, 4096).astype(np.float32), 2, 57344.0, 2.0 ** -14)
    scale_x = (0.02 * (1.0 + np.random.rand(1))).astype(np.float32)
    scale_w = (0.02 * (1.0 + np.random.rand(1))).astype(np.float32)
    return {"x": x, "w_mat": w_mat, "scale_x": scale_x, "scale_w": scale_w}

def reference(x, w_mat, scale_x, scale_w):
    acc = (x @ w_mat).astype(np.float32)
    y = (acc * (scale_x * scale_w)).astype(np.float32)
    return (y / (1.0 + np.exp(-np.clip(y, -60.0, 60.0)))).astype(np.float32)
```

```cpp
#include <hip/hip_runtime.h>
#include <rccl/rccl.h>
#include <cstdio>
#include <type_traits>

typedef int v4i __attribute__((ext_vector_type(4)));
typedef unsigned v2u __attribute__((ext_vector_type(2)));
typedef unsigned v4u __attribute__((ext_vector_type(4)));
typedef float v2f __attribute__((ext_vector_type(2)));
typedef float v4f __attribute__((ext_vector_type(4)));
typedef float v16f __attribute__((ext_vector_type(16)));
#define LDSP __attribute__((address_space(3)))
#define GLBP __attribute__((address_space(1)))

#define PACE 210
#define PACE0 110
#define NL 2
#define LDW 17
#define VPF 0
#define VPI 1
#define SPF 1
#define LPRIO 0
#define MPRIO 0
#define PPRIO 3
#define X1 2
#define NE 48
#define HX 200
#define HS 68
#define HT 130
#define TW 0
#define STAT 0
#define PG 1
#define PB 8
#define HK 270
#define HP 210
#define NPL 10
#define ZT 30
#define ZM 0
#define HE 180
#define PS 7
#define HPF 175
#define HL 360
#define SP0 1
#define RV 6
#define XSC 0
#define LDE 44
#define LDN 0
#define TWN 2
#define HQ 70
#define HALFRD 0
#define XFAKE 0
#define NOPUSH 0
#define POLLSLEEP 1
#define NMON 0
constexpr int NWAVES = 8;

constexpr unsigned CTRL_OFF = 8 * 16384;
constexpr unsigned DONE_OFF = CTRL_OFF + 128;
constexpr unsigned FLG_OFF = DONE_OFF + 1024;
constexpr unsigned ORD_OFF = FLG_OFF + 512;
constexpr unsigned ORW_OFF = ORD_OFF + 256;
constexpr unsigned LDS_BYTES = ORW_OFF + 256;
constexpr unsigned OOR = 0x40000000u;
constexpr unsigned WS_STAGE = 1u << 20;
constexpr unsigned STAGE_BYTES = 2u << 20;
constexpr unsigned WS_HELLO = WS_STAGE + 3 * STAGE_BYTES;
constexpr unsigned DEADLINE = 30000000u;

struct Params {
  unsigned rw[4];
  const char* x; char* myws; float* out; const float* sx; const float* sw;
  int rank; int pad0;
  char* peer[8];
};
static_assert(sizeof(Params) == 128, "argument layout");

__device__ __forceinline__ unsigned now() { return (unsigned)__builtin_amdgcn_s_memrealtime(); }
__device__ __forceinline__ void cbar() { asm volatile("" ::: "memory"); }
__device__ __forceinline__ void store_sys(char* p, v4u v) {
  asm volatile("global_store_dwordx4 %0, %1, off sc0 sc1\n\ts_nop 1" ::"v"(p), "v"(v) : "memory");
}

#define MAINLOOP_ASM \
      "s_cmp_eq_u32 %[nbar], 0\n" \
      "s_cbranch_scc0 LB1_%=\n" \
      "s_barrier\n" \
      "LB1_%=:\n" \
      "buffer_load_dwordx4 v[168:171], %[vx0], %[rx], %[xo] offen offset:0\n" \
      "buffer_load_dwordx4 v[172:175], %[vx0], %[rx], %[xo] offen offset:64\n" \
      "buffer_load_dwordx4 v[176:179], %[vx1], %[rx], %[xo] offen offset:0\n" \
      "buffer_load_dwordx4 v[180:183], %[vx1], %[rx], %[xo] offen offset:64\n" \
      ".if %[x1] == 0\n" \
      "buffer_load_dwordx4 v[184:187], %[vx0], %[rx], %[xo] offen offset:256\n" \
      "buffer_load_dwordx4 v[188:191], %[vx0], %[rx], %[xo] offen offset:320\n" \
      "buffer_load_dwordx4 v[192:195], %[vx1], %[rx], %[xo] offen offset:256\n" \
      "buffer_load_dwordx4 v[196:199], %[vx1], %[rx], %[xo] offen offset:320\n" \
      ".endif\n" \
      "s_cbranch_scc1 LB2_%=\n" \
      "s_barrier\n" \
      "LB2_%=:\n" \
      "s_load_dword %[ssx], %[psx], 0x0\n" \
      "s_load_dword %[ssw], %[psw], 0x0\n" \
      "v_mov_b32 v237, %[ctl]\n" \
      "v_xor_b32 v24, 0, %[ab]\n" \
      "v_or_b32 v32, 0x10000, v24\n" \
      "v_xor_b32 v25, 16, %[ab]\n" \
      "v_or_b32 v33, 0x10000, v25\n" \
      "v_xor_b32 v26, 32, %[ab]\n" \
      "v_or_b32 v34, 0x10000, v26\n" \
      "v_xor_b32 v27, 48, %[ab]\n" \
      "v_or_b32 v35, 0x10000, v27\n" \
      "v_xor_b32 v28, 64, %[ab]\n" \
      "v_or_b32 v36, 0x10000, v28\n" \
      "v_xor_b32 v29, 80, %[ab]\n" \
      "v_or_b32 v37, 0x10000, v29\n" \
      "v_xor_b32 v30, 96, %[ab]\n" \
      "v_or_b32 v38, 0x10000, v30\n" \
      "v_xor_b32 v31, 112, %[ab]\n" \
      "v_or_b32 v39, 0x10000, v31\n" \
      "v_mov_b32 v40, 0\n" \
      "v_mov_b32 v41, 0\n" \
      "v_mov_b32 v42, 0\n" \
      "v_mov_b32 v43, 0\n" \
      "v_mov_b32 v44, 0\n" \
      "v_mov_b32 v45, 0\n" \
      "v_mov_b32 v46, 0\n" \
      "v_mov_b32 v47, 0\n" \
      "v_mov_b32 v48, 0\n" \
      "v_mov_b32 v49, 0\n" \
      "v_mov_b32 v50, 0\n" \
      "v_mov_b32 v51, 0\n" \
      "v_mov_b32 v52, 0\n" \
      "v_mov_b32 v53, 0\n" \
      "v_mov_b32 v54, 0\n" \
      "v_mov_b32 v55, 0\n" \
      "v_mov_b32 v56, 0\n" \
      "v_mov_b32 v57, 0\n" \
      "v_mov_b32 v58, 0\n" \
      "v_mov_b32 v59, 0\n" \
      "v_mov_b32 v60, 0\n" \
      "v_mov_b32 v61, 0\n" \
      "v_mov_b32 v62, 0\n" \
      "v_mov_b32 v63, 0\n" \
      "v_mov_b32 v64, 0\n" \
      "v_mov_b32 v65, 0\n" \
      "v_mov_b32 v66, 0\n" \
      "v_mov_b32 v67, 0\n" \
      "v_mov_b32 v68, 0\n" \
      "v_mov_b32 v69, 0\n" \
      "v_mov_b32 v70, 0\n" \
      "v_mov_b32 v71, 0\n" \
      "v_mov_b32 v72, 0\n" \
      "v_mov_b32 v73, 0\n" \
      "v_mov_b32 v74, 0\n" \
      "v_mov_b32 v75, 0\n" \
      "v_mov_b32 v76, 0\n" \
      "v_mov_b32 v77, 0\n" \
      "v_mov_b32 v78, 0\n" \
      "v_mov_b32 v79, 0\n" \
      "v_mov_b32 v80, 0\n" \
      "v_mov_b32 v81, 0\n" \
      "v_mov_b32 v82, 0\n" \
      "v_mov_b32 v83, 0\n" \
      "v_mov_b32 v84, 0\n" \
      "v_mov_b32 v85, 0\n" \
      "v_mov_b32 v86, 0\n" \
      "v_mov_b32 v87, 0\n" \
      "v_mov_b32 v88, 0\n" \
      "v_mov_b32 v89, 0\n" \
      "v_mov_b32 v90, 0\n" \
      "v_mov_b32 v91, 0\n" \
      "v_mov_b32 v92, 0\n" \
      "v_mov_b32 v93, 0\n" \
      "v_mov_b32 v94, 0\n" \
      "v_mov_b32 v95, 0\n" \
      "v_mov_b32 v96, 0\n" \
      "v_mov_b32 v97, 0\n" \
      "v_mov_b32 v98, 0\n" \
      "v_mov_b32 v99, 0\n" \
      "v_mov_b32 v100, 0\n" \
      "v_mov_b32 v101, 0\n" \
      "v_mov_b32 v102, 0\n" \
      "v_mov_b32 v103, 0\n" \
      ".if %[x1] == 1\n" \
      "buffer_load_dwordx4 v[184:187], %[vx0], %[rx], %[xo] offen offset:256\n" \
      "buffer_load_dwordx4 v[188:191], %[vx0], %[rx], %[xo] offen offset:320\n" \
      "buffer_load_dwordx4 v[192:195], %[vx1], %[rx], %[xo] offen offset:256\n" \
      "buffer_load_dwordx4 v[196:199], %[vx1], %[rx], %[xo] offen offset:320\n" \
      ".endif\n" \
      "s_mov_b32 %[sto], 0\n" \
      "s_mov_b32 %[sj], %[par]\n" \
      "v_mov_b32 v236, %[par]\n" \
      "v_add_u32 v236, -1, v236\n" \
      "LPa_%=:\n" \
      "s_movk_i32 %[sc], 64\n" \
      "LPb_%=:\n" \
      "ds_read_b32 v234, v237 offset:8\n" \
      "s_waitcnt lgkmcnt(0)\n" \
      "s_sub_u32 %[sc], %[sc], 1\n" \
      "v_readfirstlane_b32 %[sf], v234\n" \
      "s_cmp_lg_u32 %[sf], 0\n" \
      "s_cbranch_scc1 LPc_%=\n" \
      "s_cmp_lg_u32 %[sc], 0\n" \
      "s_cbranch_scc1 LPb_%=\n" \
      "s_memrealtime vcc\n" \
      "s_waitcnt lgkmcnt(0)\n" \
      "s_sub_u32 %[sc], vcc_lo, %[t0]\n" \
      "s_cmp_lt_u32 %[sc], %[dl]\n" \
      "s_cbranch_scc1 LPa_%=\n" \
      "s_mov_b32 %[sto], 1\n" \
      "LPc_%=:\n" \
      ".if %[x1] == 2\n" \
      "buffer_load_dwordx4 v[184:187], %[vx0], %[rx], %[xo] offen offset:256\n" \
      "buffer_load_dwordx4 v[188:191], %[vx0], %[rx], %[xo] offen offset:320\n" \
      "buffer_load_dwordx4 v[192:195], %[vx1], %[rx], %[xo] offen offset:256\n" \
      "buffer_load_dwordx4 v[196:199], %[vx1], %[rx], %[xo] offen offset:320\n" \
      ".endif\n" \
      "ds_read_b64_tr_b8 v[104:105], v24 offset:0\n" \
      "ds_read_b64_tr_b8 v[106:107], v24 offset:1024\n" \
      "ds_read_b64_tr_b8 v[108:109], v24 offset:8192\n" \
      "ds_read_b64_tr_b8 v[110:111], v24 offset:9216\n" \
      "ds_read_b64_tr_b8 v[112:113], v25 offset:0\n" \
      "ds_read_b64_tr_b8 v[114:115], v25 offset:1024\n" \
      "ds_read_b64_tr_b8 v[116:117], v25 offset:8192\n" \
      "ds_read_b64_tr_b8 v[118:119], v25 offset:9216\n" \
      "ds_read_b64_tr_b8 v[120:121], v26 offset:0\n" \
      "ds_read_b64_tr_b8 v[122:123], v26 offset:1024\n" \
      "ds_read_b64_tr_b8 v[124:125], v26 offset:8192\n" \
      "ds_read_b64_tr_b8 v[126:127], v26 offset:9216\n" \
      "ds_read_b64_tr_b8 v[128:129], v27 offset:0\n" \
      "ds_read_b64_tr_b8 v[130:131], v27 offset:1024\n" \
      "ds_read_b64_tr_b8 v[132:133], v27 offset:8192\n" \
      "ds_read_b64_tr_b8 v[134:135], v27 offset:9216\n" \
      "ds_read2_b32 v[234:235], v237 offset0:0 offset1:142\n" \
      "ds_read_b64_tr_b8 v[136:137], v28 offset:0\n" \
      "ds_read_b64_tr_b8 v[138:139], v28 offset:1024\n" \
      "ds_read_b64_tr_b8 v[140:141], v28 offset:8192\n" \
      "ds_read_b64_tr_b8 v[142:143], v28 offset:9216\n" \
      "ds_read_b64_tr_b8 v[144:145], v29 offset:0\n" \
      "ds_read_b64_tr_b8 v[146:147], v29 offset:1024\n" \
      "ds_read_b64_tr_b8 v[148:149], v29 offset:8192\n" \
      "ds_read_b64_tr_b8 v[150:151], v29 offset:9216\n" \
      "ds_read_b64_tr_b8 v[152:153], v30 offset:0\n" \
      "ds_read_b64_tr_b8 v[154:155], v30 offset:1024\n" \
      "ds_read_b64_tr_b8 v[156:157], v30 offset:8192\n" \
      "ds_read_b64_tr_b8 v[158:159], v30 offset:9216\n" \
      "ds_read_b64_tr_b8 v[160:161], v31 offset:0\n" \
      "ds_read_b64_tr_b8 v[162:163], v31 offset:1024\n" \
      "s_waitcnt vmcnt(4)\n" \
      "LS0_%=:\n" \
      "s_waitcnt lgkmcnt(14)\n" \
      "v_mfma_f32_16x16x128_f8f6f4 v[40:43], v[104:111], v[168:175], v[40:43] cbsz:1\n" \
      "ds_read_b64_tr_b8 v[164:165], v31 offset:8192\n" \
      "ds_read_b64_tr_b8 v[166:167], v31 offset:9216\n" \
      "v_add_u32 v236, 2, v236\n" \
      "ds_write_b32 %[vdone], v236\n" \
      "v_add_u32 v237, -8, v237\n" \
      "v_mfma_f32_16x16x128_f8f6f4 v[72:75], v[104:111], v[176:183], v[72:75] cbsz:1\n" \
      "v_readfirstlane_b32 %[sf], v234\n" \
      "v_readfirstlane_b32 %[so2], v235\n" \
      "s_cmp_lg_u32 %[sf], 0\n" \
      "s_cbranch_scc0 LSW0_%=\n" \
      "ds_read_b64_tr_b8 v[104:105], v24 offset:32768\n" \
      "ds_read_b64_tr_b8 v[106:107], v24 offset:33792\n" \
      "v_mfma_f32_16x16x128_f8f6f4 v[44:47], v[112:119], v[168:175], v[44:47] cbsz:1\n" \
      "ds_read_b64_tr_b8 v[108:109], v24 offset:40960\n" \
      "ds_read_b64_tr_b8 v[110:111], v24 offset:41984\n" \
      ".if %[xsc] == 0\n" \
      "buffer_load_dwordx4 v[200:203], %[vs0], %[rs], %[so2] offen offset:0 sc0 sc1\n" \
      ".elseif %[xsc] == 1\n" \
      "buffer_load_dwordx4 v[200:203], %[vs0], %[rs], %[so2] offen offset:0\n" \
      ".elseif %[xsc] == 2\n" \
      "buffer_load_dwordx4 v[200:203], %[vs0], %[rs], %[so2] offen offset:0 sc0\n" \
      ".elseif %[xsc] == 3\n" \
      "buffer_load_dwordx4 v[200:203], %[vs0], %[rs], %[so2] offen offset:0 sc1\n" \
      ".else\n" \
      "buffer_load_dwordx4 v[200:203], %[vs0], %[rs], %[so2] offen offset:0 nt\n" \
      ".endif\n" \
      "v_mfma_f32_16x16x128_f8f6f4 v[76:79], v[112:119], v[176:183], v[76:79] cbsz:1\n" \
      "ds_read_b64_tr_b8 v[112:113], v25 offset:32768\n" \
      "ds_read_b64_tr_b8 v[114:115], v25 offset:33792\n" \
      "v_mfma_f32_16x16x128_f8f6f4 v[48:51], v[120:127], v[168:175], v[48:51] cbsz:1\n" \
      "ds_read_b64_tr_b8 v[116:117], v25 offset:40960\n" \
      "ds_read_b64_tr_b8 v[118:119], v25 offset:41984\n" \
      ".if %[xsc] == 0\n" \
      "buffer_load_dwordx4 v[204:207], %[vs0], %[rs], %[so2] offen offset:1024 sc0 sc1\n" \
      ".elseif %[xsc] == 1\n" \
      "buffer_load_dwordx4 v[204:207], %[vs0], %[rs], %[so2] offen offset:1024\n" \
      ".elseif %[xsc] == 2\n" \
      "buffer_load_dwordx4 v[204:207], %[vs0], %[rs], %[so2] offen offset:1024 sc0\n" \
      ".elseif %[xsc] == 3\n" \
      "buffer_load_dwordx4 v[204:207], %[vs0], %[rs], %[so2] offen offset:1024 sc1\n" \
      ".else\n" \
      "buffer_load_dwordx4 v[204:207], %[vs0], %[rs], %[so2] offen offset:1024 nt\n" \
      ".endif\n" \
      "v_mfma_f32_16x16x128_f8f6f4 v[80:83], v[120:127], v[176:183], v[80:83] cbsz:1\n" \
      "ds_read_b64_tr_b8 v[120:121], v26 offset:32768\n" \
      "ds_read_b64_tr_b8 v[122:123], v26 offset:33792\n" \
      "v_mfma_f32_16x16x128_f8f6f4 v[52:55], v[128:135], v[168:175], v[52:55] cbsz:1\n" \
      "ds_read_b64_tr_b8 v[124:125], v26 offset:40960\n" \
      "ds_read_b64_tr_b8 v[126:127], v26 offset:41984\n" \
      ".if %[xsc] == 0\n" \
      "buffer_load_dwordx4 v[208:211], %[vs1], %[rs], %[so2] offen offset:0 sc0 sc1\n" \
      ".elseif %[xsc] == 1\n" \
      "buffer_load_dwordx4 v[208:211], %[vs1], %[rs], %[so2] offen offset:0\n" \
      ".elseif %[xsc] == 2\n" \
      "buffer_load_dwordx4 v[208:211], %[vs1], %[rs], %[so2] offen offset:0 sc0\n" \
      ".elseif %[xsc] == 3\n" \
      "buffer_load_dwordx4 v[208:211], %[vs1], %[rs], %[so2] offen offset:0 sc1\n" \
      ".else\n" \
      "buffer_load_dwordx4 v[208:211], %[vs1], %[rs], %[so2] offen offset:0 nt\n" \
      ".endif\n" \
      "v_mfma_f32_16x16x128_f8f6f4 v[84:87], v[128:135], v[176:183], v[84:87] cbsz:1\n" \
      "ds_read_b64_tr_b8 v[128:129], v27 offset:32768\n" \
      "ds_read_b64_tr_b8 v[130:131], v27 offset:33792\n" \
      "s_waitcnt lgkmcnt(15)\n" \
      "v_mfma_f32_16x16x128_f8f6f4 v[56:59], v[136:143], v[168:175], v[56:59] cbsz:1\n" \
      "ds_read_b64_tr_b8 v[132:133], v27 offset:40960\n" \
      "ds_read_b64_tr_b8 v[134:135], v27 offset:41984\n" \
      "ds_read2_b32 v[234:235], v237 offset0:0 offset1:142\n" \
      ".if %[xsc] == 0\n" \
      "buffer_load_dwordx4 v[212:215], %[vs1], %[rs], %[so2] offen offset:1024 sc0 sc1\n" \
      ".elseif %[xsc] == 1\n" \
      "buffer_load_dwordx4 v[212:215], %[vs1], %[rs], %[so2] offen offset:1024\n" \
      ".elseif %[xsc] == 2\n" \
      "buffer_load_dwordx4 v[212:215], %[vs1], %[rs], %[so2] offen offset:1024 sc0\n" \
      ".elseif %[xsc] == 3\n" \
      "buffer_load_dwordx4 v[212:215], %[vs1], %[rs], %[so2] offen offset:1024 sc1\n" \
      ".else\n" \
      "buffer_load_dwordx4 v[212:215], %[vs1], %[rs], %[so2] offen offset:1024 nt\n" \
      ".endif\n" \
      "v_mfma_f32_16x16x128_f8f6f4 v[88:91], v[136:143], v[176:183], v[88:91] cbsz:1\n" \
      "ds_read_b64_tr_b8 v[136:137], v28 offset:32768\n" \
      "ds_read_b64_tr_b8 v[138:139], v28 offset:33792\n" \
      "v_mfma_f32_16x16x128_f8f6f4 v[60:63], v[144:151], v[168:175], v[60:63] cbsz:1\n" \
      "ds_read_b64_tr_b8 v[140:141], v28 offset:40960\n" \
      "ds_read_b64_tr_b8 v[142:143], v28 offset:41984\n" \
      "s_add_u32 %[sj], %[sj], 2\n" \
      "v_mfma_f32_16x16x128_f8f6f4 v[92:95], v[144:151], v[176:183], v[92:95] cbsz:1\n" \
      "ds_read_b64_tr_b8 v[144:145], v29 offset:32768\n" \
      "ds_read_b64_tr_b8 v[146:147], v29 offset:33792\n" \
      "v_mfma_f32_16x16x128_f8f6f4 v[64:67], v[152:159], v[168:175], v[64:67] cbsz:1\n" \
      "ds_read_b64_tr_b8 v[148:149], v29 offset:40960\n" \
      "ds_read_b64_tr_b8 v[150:151], v29 offset:41984\n" \
      "v_mfma_f32_16x16x128_f8f6f4 v[96:99], v[152:159], v[176:183], v[96:99] cbsz:1\n" \
      "ds_read_b64_tr_b8 v[152:153], v30 offset:32768\n" \
      "ds_read_b64_tr_b8 v[154:155], v30 offset:33792\n" \
      "v_mfma_f32_16x16x128_f8f6f4 v[68:71], v[160:167], v[168:175], v[68:71] cbsz:1\n" \
      "ds_read_b64_tr_b8 v[156:157], v30 offset:40960\n" \
      "ds_read_b64_tr_b8 v[158:159], v30 offset:41984\n" \
      "s_waitcnt vmcnt(4)\n" \
      "v_min3_u32 v232, v184, v188, v192\n" \
      "v_min_u32 v232, v232, v196\n" \
      "v_mfma_f32_16x16x128_f8f6f4 v[100:103], v[160:167], v[176:183], v[100:103] cbsz:1\n" \
      "v_cmp_eq_u32 vcc, %[xz], v232\n" \
      "ds_read_b64_tr_b8 v[160:161], v31 offset:32768\n" \
      "ds_read_b64_tr_b8 v[162:163], v31 offset:33792\n" \
      "s_cbranch_vccnz LSA1_%=\n" \
      "LS1_%=:\n" \
      "s_waitcnt lgkmcnt(14)\n" \
      "v_mfma_f32_16x16x128_f8f6f4 v[40:43], v[104:111], v[184:191], v[40:43] cbsz:1\n" \
      "ds_read_b64_tr_b8 v[164:165], v31 offset:40960\n" \
      "ds_read_b64_tr_b8 v[166:167], v31 offset:41984\n" \
      "v_add_u32 v236, 2, v236\n" \
      "ds_write_b32 %[vdone], v236\n" \
      "v_add_u32 v237, -8, v237\n" \
      "v_mfma_f32_16x16x128_f8f6f4 v[72:75], v[104:111], v[192:199], v[72:75] cbsz:1\n" \
      "v_readfirstlane_b32 %[sf], v234\n" \
      "v_readfirstlane_b32 %[so3], v235\n" \
      "s_cmp_lg_u32 %[sf], 0\n" \
      "s_cbranch_scc0 LSW1_%=\n" \
      "ds_read_b64_tr_b8 v[104:105], v32 offset:0\n" \
      "ds_read_b64_tr_b8 v[106:107], v32 offset:1024\n" \
      "v_mfma_f32_16x16x128_f8f6f4 v[44:47], v[112:119], v[184:191], v[44:47] cbsz:1\n" \
      "ds_read_b64_tr_b8 v[108:109], v32 offset:8192\n" \
      "ds_read_b64_tr_b8 v[110:111], v32 offset:9216\n" \
      ".if %[xsc] == 0\n" \
      "buffer_load_dwordx4 v[216:219], %[vs0], %[rs], %[so3] offen offset:0 sc0 sc1\n" \
      ".elseif %[xsc] == 1\n" \
      "buffer_load_dwordx4 v[216:219], %[vs0], %[rs], %[so3] offen offset:0\n" \
      ".elseif %[xsc] == 2\n" \
      "buffer_load_dwordx4 v[216:219], %[vs0], %[rs], %[so3] offen offset:0 sc0\n" \
      ".elseif %[xsc] == 3\n" \
      "buffer_load_dwordx4 v[216:219], %[vs0], %[rs], %[so3] offen offset:0 sc1\n" \
      ".else\n" \
      "buffer_load_dwordx4 v[216:219], %[vs0], %[rs], %[so3] offen offset:0 nt\n" \
      ".endif\n" \
      "v_mfma_f32_16x16x128_f8f6f4 v[76:79], v[112:119], v[192:199], v[76:79] cbsz:1\n" \
      "ds_read_b64_tr_b8 v[112:113], v33 offset:0\n" \
      "ds_read_b64_tr_b8 v[114:115], v33 offset:1024\n" \
      "v_mfma_f32_16x16x128_f8f6f4 v[48:51], v[120:127], v[184:191], v[48:51] cbsz:1\n" \
      "ds_read_b64_tr_b8 v[116:117], v33 offset:8192\n" \
      "ds_read_b64_tr_b8 v[118:119], v33 offset:9216\n" \
      ".if %[xsc] == 0\n" \
      "buffer_load_dwordx4 v[220:223], %[vs0], %[rs], %[so3] offen offset:1024 sc0 sc1\n" \
      ".elseif %[xsc] == 1\n" \
      "buffer_load_dwordx4 v[220:223], %[vs0], %[rs], %[so3] offen offset:1024\n" \
      ".elseif %[xsc] == 2\n" \
      "buffer_load_dwordx4 v[220:223], %[vs0], %[rs], %[so3] offen offset:1024 sc0\n" \
      ".elseif %[xsc] == 3\n" \
      "buffer_load_dwordx4 v[220:223], %[vs0], %[rs], %[so3] offen offset:1024 sc1\n" \
      ".else\n" \
      "buffer_load_dwordx4 v[220:223], %[vs0], %[rs], %[so3] offen offset:1024 nt\n" \
      ".endif\n" \
      "v_mfma_f32_16x16x128_f8f6f4 v[80:83], v[120:127], v[192:199], v[80:83] cbsz:1\n" \
      "ds_read_b64_tr_b8 v[120:121], v34 offset:0\n" \
      "ds_read_b64_tr_b8 v[122:123], v34 offset:1024\n" \
      "v_mfma_f32_16x16x128_f8f6f4 v[52:55], v[128:135], v[184:191], v[52:55] cbsz:1\n" \
      "ds_read_b64_tr_b8 v[124:125], v34 offset:8192\n" \
      "ds_read_b64_tr_b8 v[126:127], v34 offset:9216\n" \
      ".if %[xsc] == 0\n" \
      "buffer_load_dwordx4 v[224:227], %[vs1], %[rs], %[so3] offen offset:0 sc0 sc1\n" \
      ".elseif %[xsc] == 1\n" \
      "buffer_load_dwordx4 v[224:227], %[vs1], %[rs], %[so3] offen offset:0\n" \
      ".elseif %[xsc] == 2\n" \
      "buffer_load_dwordx4 v[224:227], %[vs1], %[rs], %[so3] offen offset:0 sc0\n" \
      ".elseif %[xsc] == 3\n" \
      "buffer_load_dwordx4 v[224:227], %[vs1], %[rs], %[so3] offen offset:0 sc1\n" \
      ".else\n" \
      "buffer_load_dwordx4 v[224:227], %[vs1], %[rs], %[so3] offen offset:0 nt\n" \
      ".endif\n" \
      "v_mfma_f32_16x16x128_f8f6f4 v[84:87], v[128:135], v[192:199], v[84:87] cbsz:1\n" \
      "ds_read_b64_tr_b8 v[128:129], v35 offset:0\n" \
      "ds_read_b64_tr_b8 v[130:131], v35 offset:1024\n" \
      "s_waitcnt lgkmcnt(15)\n" \
      "v_mfma_f32_16x16x128_f8f6f4 v[56:59], v[136:143], v[184:191], v[56:59] cbsz:1\n" \
      "ds_read_b64_tr_b8 v[132:133], v35 offset:8192\n" \
      "ds_read_b64_tr_b8 v[134:135], v35 offset:9216\n" \
      "ds_read2_b32 v[234:235], v237 offset0:0 offset1:142\n" \
      ".if %[xsc] == 0\n" \
      "buffer_load_dwordx4 v[228:231], %[vs1], %[rs], %[so3] offen offset:1024 sc0 sc1\n" \
      ".elseif %[xsc] == 1\n" \
      "buffer_load_dwordx4 v[228:231], %[vs1], %[rs], %[so3] offen offset:1024\n" \
      ".elseif %[xsc] == 2\n" \
      "buffer_load_dwordx4 v[228:231], %[vs1], %[rs], %[so3] offen offset:1024 sc0\n" \
      ".elseif %[xsc] == 3\n" \
      "buffer_load_dwordx4 v[228:231], %[vs1], %[rs], %[so3] offen offset:1024 sc1\n" \
      ".else\n" \
      "buffer_load_dwordx4 v[228:231], %[vs1], %[rs], %[so3] offen offset:1024 nt\n" \
      ".endif\n" \
      "v_mfma_f32_16x16x128_f8f6f4 v[88:91], v[136:143], v[192:199], v[88:91] cbsz:1\n" \
      "ds_read_b64_tr_b8 v[136:137], v36 offset:0\n" \
      "ds_read_b64_tr_b8 v[138:139], v36 offset:1024\n" \
      "v_mfma_f32_16x16x128_f8f6f4 v[60:63], v[144:151], v[184:191], v[60:63] cbsz:1\n" \
      "ds_read_b64_tr_b8 v[140:141], v36 offset:8192\n" \
      "ds_read_b64_tr_b8 v[142:143], v36 offset:9216\n" \
      "s_add_u32 %[sj], %[sj], 2\n" \
      "v_mfma_f32_16x16x128_f8f6f4 v[92:95], v[144:151], v[192:199], v[92:95] cbsz:1\n" \
      "ds_read_b64_tr_b8 v[144:145], v37 offset:0\n" \
      "ds_read_b64_tr_b8 v[146:147], v37 offset:1024\n" \
      "v_mfma_f32_16x16x128_f8f6f4 v[64:67], v[152:159], v[184:191], v[64:67] cbsz:1\n" \
      "ds_read_b64_tr_b8 v[148:149], v37 offset:8192\n" \
      "ds_read_b64_tr_b8 v[150:151], v37 offset:9216\n" \
      "v_mfma_f32_16x16x128_f8f6f4 v[96:99], v[152:159], v[192:199], v[96:99] cbsz:1\n" \
      "ds_read_b64_tr_b8 v[152:153], v38 offset:0\n" \
      "ds_read_b64_tr_b8 v[154:155], v38 offset:1024\n" \
      "v_mfma_f32_16x16x128_f8f6f4 v[68:71], v[160:167], v[184:191], v[68:71] cbsz:1\n" \
      "ds_read_b64_tr_b8 v[156:157], v38 offset:8192\n" \
      "ds_read_b64_tr_b8 v[158:159], v38 offset:9216\n" \
      "s_waitcnt vmcnt(4)\n" \
      "v_min3_u32 v232, v200, v204, v208\n" \
      "v_min_u32 v232, v232, v212\n" \
      "v_mfma_f32_16x16x128_f8f6f4 v[100:103], v[160:167], v[192:199], v[100:103] cbsz:1\n" \
      "v_cmp_eq_u32 vcc, %[xz], v232\n" \
      "ds_read_b64_tr_b8 v[160:161], v39 offset:0\n" \
      "ds_read_b64_tr_b8 v[162:163], v39 offset:1024\n" \
      "s_cbranch_vccnz LSA2_%=\n" \
      "LS2_%=:\n" \
      "s_waitcnt lgkmcnt(14)\n" \
      "v_mfma_f32_16x16x128_f8f6f4 v[40:43], v[104:111], v[200:207], v[40:43] cbsz:1\n" \
      "ds_read_b64_tr_b8 v[164:165], v39 offset:8192\n" \
      "ds_read_b64_tr_b8 v[166:167], v39 offset:9216\n" \
      "v_add_u32 v236, 2, v236\n" \
      "ds_write_b32 %[vdone], v236\n" \
      "v_add_u32 v237, -8, v237\n" \
      "v_mfma_f32_16x16x128_f8f6f4 v[72:75], v[104:111], v[208:215], v[72:75] cbsz:1\n" \
      "v_readfirstlane_b32 %[sf], v234\n" \
      "v_readfirstlane_b32 %[so0], v235\n" \
      "s_cmp_lg_u32 %[sf], 0\n" \
      "s_cbranch_scc0 LSW2_%=\n" \
      "ds_read_b64_tr_b8 v[104:105], v32 offset:32768\n" \
      "ds_read_b64_tr_b8 v[106:107], v32 offset:33792\n" \
      "v_mfma_f32_16x16x128_f8f6f4 v[44:47], v[112:119], v[200:207], v[44:47] cbsz:1\n" \
      "ds_read_b64_tr_b8 v[108:109], v32 offset:40960\n" \
      "ds_read_b64_tr_b8 v[110:111], v32 offset:41984\n" \
      ".if %[xsc] == 0\n" \
      "buffer_load_dwordx4 v[168:171], %[vs0], %[rs], %[so0] offen offset:0 sc0 sc1\n" \
      ".elseif %[xsc] == 1\n" \
      "buffer_load_dwordx4 v[168:171], %[vs0], %[rs], %[so0] offen offset:0\n" \
      ".elseif %[xsc] == 2\n" \
      "buffer_load_dwordx4 v[168:171], %[vs0], %[rs], %[so0] offen offset:0 sc0\n" \
      ".elseif %[xsc] == 3\n" \
      "buffer_load_dwordx4 v[168:171], %[vs0], %[rs], %[so0] offen offset:0 sc1\n" \
      ".else\n" \
      "buffer_load_dwordx4 v[168:171], %[vs0], %[rs], %[so0] offen offset:0 nt\n" \
      ".endif\n" \
      "v_mfma_f32_16x16x128_f8f6f4 v[76:79], v[112:119], v[208:215], v[76:79] cbsz:1\n" \
      "ds_read_b64_tr_b8 v[112:113], v33 offset:32768\n" \
      "ds_read_b64_tr_b8 v[114:115], v33 offset:33792\n" \
      "v_mfma_f32_16x16x128_f8f6f4 v[48:51], v[120:127], v[200:207], v[48:51] cbsz:1\n" \
      "ds_read_b64_tr_b8 v[116:117], v33 offset:40960\n" \
      "ds_read_b64_tr_b8 v[118:119], v33 offset:41984\n" \
      ".if %[xsc] == 0\n" \
      "buffer_load_dwordx4 v[172:175], %[vs0], %[rs], %[so0] offen offset:1024 sc0 sc1\n" \
      ".elseif %[xsc] == 1\n" \
      "buffer_load_dwordx4 v[172:175], %[vs0], %[rs], %[so0] offen offset:1024\n" \
      ".elseif %[xsc] == 2\n" \
      "buffer_load_dwordx4 v[172:175], %[vs0], %[rs], %[so0] offen offset:1024 sc0\n" \
      ".elseif %[xsc] == 3\n" \
      "buffer_load_dwordx4 v[172:175], %[vs0], %[rs], %[so0] offen offset:1024 sc1\n" \
      ".else\n" \
      "buffer_load_dwordx4 v[172:175], %[vs0], %[rs], %[so0] offen offset:1024 nt\n" \
      ".endif\n" \
      "v_mfma_f32_16x16x128_f8f6f4 v[80:83], v[120:127], v[208:215], v[80:83] cbsz:1\n" \
      "ds_read_b64_tr_b8 v[120:121], v34 offset:32768\n" \
      "ds_read_b64_tr_b8 v[122:123], v34 offset:33792\n" \
      "v_mfma_f32_16x16x128_f8f6f4 v[52:55], v[128:135], v[200:207], v[52:55] cbsz:1\n" \
      "ds_read_b64_tr_b8 v[124:125], v34 offset:40960\n" \
      "ds_read_b64_tr_b8 v[126:127], v34 offset:41984\n" \
      ".if %[xsc] == 0\n" \
      "buffer_load_dwordx4 v[176:179], %[vs1], %[rs], %[so0] offen offset:0 sc0 sc1\n" \
      ".elseif %[xsc] == 1\n" \
      "buffer_load_dwordx4 v[176:179], %[vs1], %[rs], %[so0] offen offset:0\n" \
      ".elseif %[xsc] == 2\n" \
      "buffer_load_dwordx4 v[176:179], %[vs1], %[rs], %[so0] offen offset:0 sc0\n" \
      ".elseif %[xsc] == 3\n" \
      "buffer_load_dwordx4 v[176:179], %[vs1], %[rs], %[so0] offen offset:0 sc1\n" \
      ".else\n" \
      "buffer_load_dwordx4 v[176:179], %[vs1], %[rs], %[so0] offen offset:0 nt\n" \
      ".endif\n" \
      "v_mfma_f32_16x16x128_f8f6f4 v[84:87], v[128:135], v[208:215], v[84:87] cbsz:1\n" \
      "ds_read_b64_tr_b8 v[128:129], v35 offset:32768\n" \
      "ds_read_b64_tr_b8 v[130:131], v35 offset:33792\n" \
      "s_waitcnt lgkmcnt(15)\n" \
      "v_mfma_f32_16x16x128_f8f6f4 v[56:59], v[136:143], v[200:207], v[56:59] cbsz:1\n" \
      "ds_read_b64_tr_b8 v[132:133], v35 offset:40960\n" \
      "ds_read_b64_tr_b8 v[134:135], v35 offset:41984\n" \
      "ds_read2_b32 v[234:235], v237 offset0:0 offset1:142\n" \
      ".if %[xsc] == 0\n" \
      "buffer_load_dwordx4 v[180:183], %[vs1], %[rs], %[so0] offen offset:1024 sc0 sc1\n" \
      ".elseif %[xsc] == 1\n" \
      "buffer_load_dwordx4 v[180:183], %[vs1], %[rs], %[so0] offen offset:1024\n" \
      ".elseif %[xsc] == 2\n" \
      "buffer_load_dwordx4 v[180:183], %[vs1], %[rs], %[so0] offen offset:1024 sc0\n" \
      ".elseif %[xsc] == 3\n" \
      "buffer_load_dwordx4 v[180:183], %[vs1], %[rs], %[so0] offen offset:1024 sc1\n" \
      ".else\n" \
      "buffer_load_dwordx4 v[180:183], %[vs1], %[rs], %[so0] offen offset:1024 nt\n" \
      ".endif\n" \
      "v_mfma_f32_16x16x128_f8f6f4 v[88:91], v[136:143], v[208:215], v[88:91] cbsz:1\n" \
      "ds_read_b64_tr_b8 v[136:137], v36 offset:32768\n" \
      "ds_read_b64_tr_b8 v[138:139], v36 offset:33792\n" \
      "v_mfma_f32_16x16x128_f8f6f4 v[60:63], v[144:151], v[200:207], v[60:63] cbsz:1\n" \
      "ds_read_b64_tr_b8 v[140:141], v36 offset:40960\n" \
      "ds_read_b64_tr_b8 v[142:143], v36 offset:41984\n" \
      "s_add_u32 %[sj], %[sj], 2\n" \
      "v_mfma_f32_16x16x128_f8f6f4 v[92:95], v[144:151], v[208:215], v[92:95] cbsz:1\n" \
      "ds_read_b64_tr_b8 v[144:145], v37 offset:32768\n" \
      "ds_read_b64_tr_b8 v[146:147], v37 offset:33792\n" \
      "v_mfma_f32_16x16x128_f8f6f4 v[64:67], v[152:159], v[200:207], v[64:67] cbsz:1\n" \
      "ds_read_b64_tr_b8 v[148:149], v37 offset:40960\n" \
      "ds_read_b64_tr_b8 v[150:151], v37 offset:41984\n" \
      "v_mfma_f32_16x16x128_f8f6f4 v[96:99], v[152:159], v[208:215], v[96:99] cbsz:1\n" \
      "ds_read_b64_tr_b8 v[152:153], v38 offset:32768\n" \
      "ds_read_b64_tr_b8 v[154:155], v38 offset:33792\n" \
      "v_mfma_f32_16x16x128_f8f6f4 v[68:71], v[160:167], v[200:207], v[68:71] cbsz:1\n" \
      "ds_read_b64_tr_b8 v[156:157], v38 offset:40960\n" \
      "ds_read_b64_tr_b8 v[158:159], v38 offset:41984\n" \
      "s_waitcnt vmcnt(4)\n" \
      "v_min3_u32 v232, v216, v220, v224\n" \
      "v_min_u32 v232, v232, v228\n" \
      "v_mfma_f32_16x16x128_f8f6f4 v[100:103], v[160:167], v[208:215], v[100:103] cbsz:1\n" \
      "v_cmp_eq_u32 vcc, %[xz], v232\n" \
      "ds_read_b64_tr_b8 v[160:161], v39 offset:32768\n" \
      "ds_read_b64_tr_b8 v[162:163], v39 offset:33792\n" \
      "s_cbranch_vccnz LSA3_%=\n" \
      "LS3_%=:\n" \
      "s_waitcnt lgkmcnt(14)\n" \
      "v_mfma_f32_16x16x128_f8f6f4 v[40:43], v[104:111], v[216:223], v[40:43] cbsz:1\n" \
      "ds_read_b64_tr_b8 v[164:165], v39 offset:40960\n" \
      "ds_read_b64_tr_b8 v[166:167], v39 offset:41984\n" \
      "v_add_u32 v236, 2, v236\n" \
      "ds_write_b32 %[vdone], v236\n" \
      "v_add_u32 v237, -8, v237\n" \
      "v_mfma_f32_16x16x128_f8f6f4 v[72:75], v[104:111], v[224:231], v[72:75] cbsz:1\n" \
      "v_readfirstlane_b32 %[sf], v234\n" \
      "v_readfirstlane_b32 %[so1], v235\n" \
      "s_cmp_ge_u32 %[sj], 30\n" \
      "s_cbranch_scc1 LT_%=\n" \
      "s_cmp_lg_u32 %[sf], 0\n" \
      "s_cbranch_scc0 LSW3_%=\n" \
      "ds_read_b64_tr_b8 v[104:105], v24 offset:0\n" \
      "ds_read_b64_tr_b8 v[106:107], v24 offset:1024\n" \
      "v_mfma_f32_16x16x128_f8f6f4 v[44:47], v[112:119], v[216:223], v[44:47] cbsz:1\n" \
      "ds_read_b64_tr_b8 v[108:109], v24 offset:8192\n" \
      "ds_read_b64_tr_b8 v[110:111], v24 offset:9216\n" \
      ".if %[xsc] == 0\n" \
      "buffer_load_dwordx4 v[184:187], %[vs0], %[rs], %[so1] offen offset:0 sc0 sc1\n" \
      ".elseif %[xsc] == 1\n" \
      "buffer_load_dwordx4 v[184:187], %[vs0], %[rs], %[so1] offen offset:0\n" \
      ".elseif %[xsc] == 2\n" \
      "buffer_load_dwordx4 v[184:187], %[vs0], %[rs], %[so1] offen offset:0 sc0\n" \
      ".elseif %[xsc] == 3\n" \
      "buffer_load_dwordx4 v[184:187], %[vs0], %[rs], %[so1] offen offset:0 sc1\n" \
      ".else\n" \
      "buffer_load_dwordx4 v[184:187], %[vs0], %[rs], %[so1] offen offset:0 nt\n" \
      ".endif\n" \
      "v_mfma_f32_16x16x128_f8f6f4 v[76:79], v[112:119], v[224:231], v[76:79] cbsz:1\n" \
      "ds_read_b64_tr_b8 v[112:113], v25 offset:0\n" \
      "ds_read_b64_tr_b8 v[114:115], v25 offset:1024\n" \
      "v_mfma_f32_16x16x128_f8f6f4 v[48:51], v[120:127], v[216:223], v[48:51] cbsz:1\n" \
      "ds_read_b64_tr_b8 v[116:117], v25 offset:8192\n" \
      "ds_read_b64_tr_b8 v[118:119], v25 offset:9216\n" \
      ".if %[xsc] == 0\n" \
      "buffer_load_dwordx4 v[188:191], %[vs0], %[rs], %[so1] offen offset:1024 sc0 sc1\n" \
      ".elseif %[xsc] == 1\n" \
      "buffer_load_dwordx4 v[188:191], %[vs0], %[rs], %[so1] offen offset:1024\n" \
      ".elseif %[xsc] == 2\n" \
      "buffer_load_dwordx4 v[188:191], %[vs0], %[rs], %[so1] offen offset:1024 sc0\n" \
      ".elseif %[xsc] == 3\n" \
      "buffer_load_dwordx4 v[188:191], %[vs0], %[rs], %[so1] offen offset:1024 sc1\n" \
      ".else\n" \
      "buffer_load_dwordx4 v[188:191], %[vs0], %[rs], %[so1] offen offset:1024 nt\n" \
      ".endif\n" \
      "v_mfma_f32_16x16x128_f8f6f4 v[80:83], v[120:127], v[224:231], v[80:83] cbsz:1\n" \
      "ds_read_b64_tr_b8 v[120:121], v26 offset:0\n" \
      "ds_read_b64_tr_b8 v[122:123], v26 offset:1024\n" \
      "v_mfma_f32_16x16x128_f8f6f4 v[52:55], v[128:135], v[216:223], v[52:55] cbsz:1\n" \
      "ds_read_b64_tr_b8 v[124:125], v26 offset:8192\n" \
      "ds_read_b64_tr_b8 v[126:127], v26 offset:9216\n" \
      ".if %[xsc] == 0\n" \
      "buffer_load_dwordx4 v[192:195], %[vs1], %[rs], %[so1] offen offset:0 sc0 sc1\n" \
      ".elseif %[xsc] == 1\n" \
      "buffer_load_dwordx4 v[192:195], %[vs1], %[rs], %[so1] offen offset:0\n" \
      ".elseif %[xsc] == 2\n" \
      "buffer_load_dwordx4 v[192:195], %[vs1], %[rs], %[so1] offen offset:0 sc0\n" \
      ".elseif %[xsc] == 3\n" \
      "buffer_load_dwordx4 v[192:195], %[vs1], %[rs], %[so1] offen offset:0 sc1\n" \
      ".else\n" \
      "buffer_load_dwordx4 v[192:195], %[vs1], %[rs], %[so1] offen offset:0 nt\n" \
      ".endif\n" \
      "v_mfma_f32_16x16x128_f8f6f4 v[84:87], v[128:135], v[224:231], v[84:87] cbsz:1\n" \
      "ds_read_b64_tr_b8 v[128:129], v27 offset:0\n" \
      "ds_read_b64_tr_b8 v[130:131], v27 offset:1024\n" \
      "s_waitcnt lgkmcnt(15)\n" \
      "v_mfma_f32_16x16x128_f8f6f4 v[56:59], v[136:143], v[216:223], v[56:59] cbsz:1\n" \
      "ds_read_b64_tr_b8 v[132:133], v27 offset:8192\n" \
      "ds_read_b64_tr_b8 v[134:135], v27 offset:9216\n" \
      "ds_read2_b32 v[234:235], v237 offset0:0 offset1:142\n" \
      ".if %[xsc] == 0\n" \
      "buffer_load_dwordx4 v[196:199], %[vs1], %[rs], %[so1] offen offset:1024 sc0 sc1\n" \
      ".elseif %[xsc] == 1\n" \
      "buffer_load_dwordx4 v[196:199], %[vs1], %[rs], %[so1] offen offset:1024\n" \
      ".elseif %[xsc] == 2\n" \
      "buffer_load_dwordx4 v[196:199], %[vs1], %[rs], %[so1] offen offset:1024 sc0\n" \
      ".elseif %[xsc] == 3\n" \
      "buffer_load_dwordx4 v[196:199], %[vs1], %[rs], %[so1] offen offset:1024 sc1\n" \
      ".else\n" \
      "buffer_load_dwordx4 v[196:199], %[vs1], %[rs], %[so1] offen offset:1024 nt\n" \
      ".endif\n" \
      "v_mfma_f32_16x16x128_f8f6f4 v[88:91], v[136:143], v[224:231], v[88:91] cbsz:1\n" \
      "ds_read_b64_tr_b8 v[136:137], v28 offset:0\n" \
      "ds_read_b64_tr_b8 v[138:139], v28 offset:1024\n" \
      "v_mfma_f32_16x16x128_f8f6f4 v[60:63], v[144:151], v[216:223], v[60:63] cbsz:1\n" \
      "ds_read_b64_tr_b8 v[140:141], v28 offset:8192\n" \
      "ds_read_b64_tr_b8 v[142:143], v28 offset:9216\n" \
      "s_add_u32 %[sj], %[sj], 2\n" \
      "v_mfma_f32_16x16x128_f8f6f4 v[92:95], v[144:151], v[224:231], v[92:95] cbsz:1\n" \
      "ds_read_b64_tr_b8 v[144:145], v29 offset:0\n" \
      "ds_read_b64_tr_b8 v[146:147], v29 offset:1024\n" \
      "v_mfma_f32_16x16x128_f8f6f4 v[64:67], v[152:159], v[216:223], v[64:67] cbsz:1\n" \
      "ds_read_b64_tr_b8 v[148:149], v29 offset:8192\n" \
      "ds_read_b64_tr_b8 v[150:151], v29 offset:9216\n" \
      "v_mfma_f32_16x16x128_f8f6f4 v[96:99], v[152:159], v[224:231], v[96:99] cbsz:1\n" \
      "ds_read_b64_tr_b8 v[152:153], v30 offset:0\n" \
      "ds_read_b64_tr_b8 v[154:155], v30 offset:1024\n" \
      "v_mfma_f32_16x16x128_f8f6f4 v[68:71], v[160:167], v[216:223], v[68:71] cbsz:1\n" \
      "ds_read_b64_tr_b8 v[156:157], v30 offset:8192\n" \
      "ds_read_b64_tr_b8 v[158:159], v30 offset:9216\n" \
      "s_waitcnt vmcnt(4)\n" \
      "v_min3_u32 v232, v168, v172, v176\n" \
      "v_min_u32 v232, v232, v180\n" \
      "v_mfma_f32_16x16x128_f8f6f4 v[100:103], v[160:167], v[224:231], v[100:103] cbsz:1\n" \
      "v_cmp_eq_u32 vcc, %[xz], v232\n" \
      "ds_read_b64_tr_b8 v[160:161], v31 offset:0\n" \
      "ds_read_b64_tr_b8 v[162:163], v31 offset:1024\n" \
      "s_cbranch_vccnz LSA0_%=\n" \
      "s_branch LS0_%=\n" \
      "LSW0_%=:\n" \
      "s_nop 2\n" \
      "v_mfma_f32_16x16x128_f8f6f4 v[44:47], v[112:119], v[168:175], v[44:47] cbsz:1\n" \
      ".if %[xsc] == 0\n" \
      "buffer_load_dwordx4 v[200:203], %[vs0], %[rs], %[so2] offen offset:0 sc0 sc1\n" \
      ".elseif %[xsc] == 1\n" \
      "buffer_load_dwordx4 v[200:203], %[vs0], %[rs], %[so2] offen offset:0\n" \
      ".elseif %[xsc] == 2\n" \
      "buffer_load_dwordx4 v[200:203], %[vs0], %[rs], %[so2] offen offset:0 sc0\n" \
      ".elseif %[xsc] == 3\n" \
      "buffer_load_dwordx4 v[200:203], %[vs0], %[rs], %[so2] offen offset:0 sc1\n" \
      ".else\n" \
      "buffer_load_dwordx4 v[200:203], %[vs0], %[rs], %[so2] offen offset:0 nt\n" \
      ".endif\n" \
      "v_mfma_f32_16x16x128_f8f6f4 v[76:79], v[112:119], v[176:183], v[76:79] cbsz:1\n" \
      "v_mfma_f32_16x16x128_f8f6f4 v[48:51], v[120:127], v[168:175], v[48:51] cbsz:1\n" \
      ".if %[xsc] == 0\n" \
      "buffer_load_dwordx4 v[204:207], %[vs0], %[rs], %[so2] offen offset:1024 sc0 sc1\n" \
      ".elseif %[xsc] == 1\n" \
      "buffer_load_dwordx4 v[204:207], %[vs0], %[rs], %[so2] offen offset:1024\n" \
      ".elseif %[xsc] == 2\n" \
      "buffer_load_dwordx4 v[204:207], %[vs0], %[rs], %[so2] offen offset:1024 sc0\n" \
      ".elseif %[xsc] == 3\n" \
      "buffer_load_dwordx4 v[204:207], %[vs0], %[rs], %[so2] offen offset:1024 sc1\n" \
      ".else\n" \
      "buffer_load_dwordx4 v[204:207], %[vs0], %[rs], %[so2] offen offset:1024 nt\n" \
      ".endif\n" \
      "v_mfma_f32_16x16x128_f8f6f4 v[80:83], v[120:127], v[176:183], v[80:83] cbsz:1\n" \
      "v_mfma_f32_16x16x128_f8f6f4 v[52:55], v[128:135], v[168:175], v[52:55] cbsz:1\n" \
      ".if %[xsc] == 0\n" \
      "buffer_load_dwordx4 v[208:211], %[vs1], %[rs], %[so2] offen offset:0 sc0 sc1\n" \
      ".elseif %[xsc] == 1\n" \
      "buffer_load_dwordx4 v[208:211], %[vs1], %[rs], %[so2] offen offset:0\n" \
      ".elseif %[xsc] == 2\n" \
      "buffer_load_dwordx4 v[208:211], %[vs1], %[rs], %[so2] offen offset:0 sc0\n" \
      ".elseif %[xsc] == 3\n" \
      "buffer_load_dwordx4 v[208:211], %[vs1], %[rs], %[so2] offen offset:0 sc1\n" \
      ".else\n" \
      "buffer_load_dwordx4 v[208:211], %[vs1], %[rs], %[so2] offen offset:0 nt\n" \
      ".endif\n" \
      "v_mfma_f32_16x16x128_f8f6f4 v[84:87], v[128:135], v[176:183], v[84:87] cbsz:1\n" \
      "s_waitcnt lgkmcnt(0)\n" \
      "v_mfma_f32_16x16x128_f8f6f4 v[56:59], v[136:143], v[168:175], v[56:59] cbsz:1\n" \
      ".if %[xsc] == 0\n" \
      "buffer_load_dwordx4 v[212:215], %[vs1], %[rs], %[so2] offen offset:1024 sc0 sc1\n" \
      ".elseif %[xsc] == 1\n" \
      "buffer_load_dwordx4 v[212:215], %[vs1], %[rs], %[so2] offen offset:1024\n" \
      ".elseif %[xsc] == 2\n" \
      "buffer_load_dwordx4 v[212:215], %[vs1], %[rs], %[so2] offen offset:1024 sc0\n" \
      ".elseif %[xsc] == 3\n" \
      "buffer_load_dwordx4 v[212:215], %[vs1], %[rs], %[so2] offen offset:1024 sc1\n" \
      ".else\n" \
      "buffer_load_dwordx4 v[212:215], %[vs1], %[rs], %[so2] offen offset:1024 nt\n" \
      ".endif\n" \
      "v_mfma_f32_16x16x128_f8f6f4 v[88:91], v[136:143], v[176:183], v[88:91] cbsz:1\n" \
      "v_mfma_f32_16x16x128_f8f6f4 v[60:63], v[144:151], v[168:175], v[60:63] cbsz:1\n" \
      "s_add_u32 %[sj], %[sj], 2\n" \
      "v_mfma_f32_16x16x128_f8f6f4 v[92:95], v[144:151], v[176:183], v[92:95] cbsz:1\n" \
      "v_mfma_f32_16x16x128_f8f6f4 v[64:67], v[152:159], v[168:175], v[64:67] cbsz:1\n" \
      "v_mfma_f32_16x16x128_f8f6f4 v[96:99], v[152:159], v[176:183], v[96:99] cbsz:1\n" \
      "v_mfma_f32_16x16x128_f8f6f4 v[68:71], v[160:167], v[168:175], v[68:71] cbsz:1\n" \
      "v_mfma_f32_16x16x128_f8f6f4 v[100:103], v[160:167], v[176:183], v[100:103] cbsz:1\n" \
      "s_mov_b32 %[sn], 0\n" \
      "s_waitcnt vmcnt(4)\n" \
      "v_min3_u32 v232, v184, v188, v192\n" \
      "v_min_u32 v232, v232, v196\n" \
      "v_cmp_eq_u32 vcc, %[xz], v232\n" \
      "s_cbranch_vccz LV0_%=\n" \
      "s_cmp_lt_u32 %[sj], 4\n" \
      "s_cbranch_scc1 LV0_%=\n" \
      "ds_read_b32 v238, v237 offset:584\n" \
      "s_mov_b32 %[sn], 1\n" \
      "s_waitcnt lgkmcnt(0)\n" \
      "v_readfirstlane_b32 %[so1], v238\n" \
      "s_mov_b64 exec, vcc\n" \
      "s_nop 3\n" \
      "buffer_load_dwordx4 v[184:187], %[vs0], %[rs], %[so1] offen offset:0 sc0 sc1\n" \
      "buffer_load_dwordx4 v[188:191], %[vs0], %[rs], %[so1] offen offset:1024 sc0 sc1\n" \
      "buffer_load_dwordx4 v[192:195], %[vs1], %[rs], %[so1] offen offset:0 sc0 sc1\n" \
      "buffer_load_dwordx4 v[196:199], %[vs1], %[rs], %[so1] offen offset:1024 sc0 sc1\n" \
      "s_mov_b64 exec, -1\n" \
      "LV0_%=:\n" \
      "LW0a_%=:\n" \
      "s_movk_i32 %[sc], 64\n" \
      "LW0b_%=:\n" \
      "ds_read_b32 v234, v237 offset:8\n" \
      "s_waitcnt lgkmcnt(0)\n" \
      "s_sub_u32 %[sc], %[sc], 1\n" \
      "v_readfirstlane_b32 %[sf], v234\n" \
      "s_cmp_lg_u32 %[sf], 0\n" \
      "s_cbranch_scc1 LW0c_%=\n" \
      "s_cmp_lg_u32 %[sc], 0\n" \
      "s_cbranch_scc1 LW0b_%=\n" \
      "s_memrealtime vcc\n" \
      "s_waitcnt lgkmcnt(0)\n" \
      "s_sub_u32 %[sc], vcc_lo, %[t0]\n" \
      "s_cmp_lt_u32 %[sc], %[dl]\n" \
      "s_cbranch_scc1 LW0a_%=\n" \
      "s_mov_b32 %[sto], 1\n" \
      "LW0c_%=:\n" \
      "ds_read_b64_tr_b8 v[104:105], v24 offset:32768\n" \
      "ds_read_b64_tr_b8 v[106:107], v24 offset:33792\n" \
      "ds_read_b64_tr_b8 v[108:109], v24 offset:40960\n" \
      "ds_read_b64_tr_b8 v[110:111], v24 offset:41984\n" \
      "ds_read_b64_tr_b8 v[112:113], v25 offset:32768\n" \
      "ds_read_b64_tr_b8 v[114:115], v25 offset:33792\n" \
      "ds_read_b64_tr_b8 v[116:117], v25 offset:40960\n" \
      "ds_read_b64_tr_b8 v[118:119], v25 offset:41984\n" \
      "ds_read_b64_tr_b8 v[120:121], v26 offset:32768\n" \
      "ds_read_b64_tr_b8 v[122:123], v26 offset:33792\n" \
      "ds_read_b64_tr_b8 v[124:125], v26 offset:40960\n" \
      "ds_read_b64_tr_b8 v[126:127], v26 offset:41984\n" \
      "ds_read_b64_tr_b8 v[128:129], v27 offset:32768\n" \
      "ds_read_b64_tr_b8 v[130:131], v27 offset:33792\n" \
      "ds_read_b64_tr_b8 v[132:133], v27 offset:40960\n" \
      "ds_read_b64_tr_b8 v[134:135], v27 offset:41984\n" \
      "ds_read2_b32 v[234:235], v237 offset0:0 offset1:142\n" \
      "ds_read_b64_tr_b8 v[136:137], v28 offset:32768\n" \
      "ds_read_b64_tr_b8 v[138:139], v28 offset:33792\n" \
      "ds_read_b64_tr_b8 v[140:141], v28 offset:40960\n" \
      "ds_read_b64_tr_b8 v[142:143], v28 offset:41984\n" \
      "ds_read_b64_tr_b8 v[144:145], v29 offset:32768\n" \
      "ds_read_b64_tr_b8 v[146:147], v29 offset:33792\n" \
      "ds_read_b64_tr_b8 v[148:149], v29 offset:40960\n" \
      "ds_read_b64_tr_b8 v[150:151], v29 offset:41984\n" \
      "ds_read_b64_tr_b8 v[152:153], v30 offset:32768\n" \
      "ds_read_b64_tr_b8 v[154:155], v30 offset:33792\n" \
      "ds_read_b64_tr_b8 v[156:157], v30 offset:40960\n" \
      "ds_read_b64_tr_b8 v[158:159], v30 offset:41984\n" \
      "ds_read_b64_tr_b8 v[160:161], v31 offset:32768\n" \
      "ds_read_b64_tr_b8 v[162:163], v31 offset:33792\n" \
      "s_cmp_eq_u32 %[sn], 0\n" \
      "s_cbranch_scc1 LS1_%=\n" \
      "s_waitcnt vmcnt(0)\n" \
      "v_min3_u32 v232, v184, v188, v192\n" \
      "v_min_u32 v232, v232, v196\n" \
      "v_cmp_eq_u32 vcc, %[xz], v232\n" \
      "s_cbranch_vccz LSB1_%=\n" \
      "LSA1_%=:\n" \
      "s_cmp_lt_u32 %[sj], 4\n" \
      "s_cbranch_scc1 LS1_%=\n" \
      "s_cmp_lt_u32 %[so1], 0x40000000\n" \
      "s_cbranch_scc1 LSP1_%=\n" \
      "LO1a_%=:\n" \
      "s_movk_i32 %[sc], 64\n" \
      "LO1b_%=:\n" \
      "ds_read_b32 v238, v237 offset:584\n" \
      "s_waitcnt lgkmcnt(0)\n" \
      "s_sub_u32 %[sc], %[sc], 1\n" \
      "v_readfirstlane_b32 %[sf], v238\n" \
      "s_cmp_lt_u32 %[sf], 0x40000000\n" \
      "s_cbranch_scc1 LO1c_%=\n" \
      "s_cmp_lg_u32 %[sc], 0\n" \
      "s_cbranch_scc1 LO1b_%=\n" \
      "s_memrealtime vcc\n" \
      "s_waitcnt lgkmcnt(0)\n" \
      "s_sub_u32 %[sc], vcc_lo, %[t0]\n" \
      "s_cmp_lt_u32 %[sc], %[dl]\n" \
      "s_cbranch_scc1 LO1a_%=\n" \
      "s_mov_b32 %[sto], 1\n" \
      "LO1c_%=:\n" \
      "s_mov_b32 %[so1], %[sf]\n" \
      "s_mov_b64 vcc, -1\n" \
      "LSP1_%=:\n" \
      "s_mov_b64 exec, vcc\n" \
      "buffer_load_dwordx4 v[184:187], %[vs0], %[rs], %[so1] offen offset:0 sc0 sc1\n" \
      "buffer_load_dwordx4 v[188:191], %[vs0], %[rs], %[so1] offen offset:1024 sc0 sc1\n" \
      "buffer_load_dwordx4 v[192:195], %[vs1], %[rs], %[so1] offen offset:0 sc0 sc1\n" \
      "buffer_load_dwordx4 v[196:199], %[vs1], %[rs], %[so1] offen offset:1024 sc0 sc1\n" \
      "s_mov_b64 exec, -1\n" \
      ".if %[npl]\n" \
      "s_cmp_lt_u32 %[so2], 0x40000000\n" \
      "s_cbranch_scc1 LSQ1_%=\n" \
      "s_movk_i32 %[sc], %[npl]\n" \
      "LSR1_%=:\n" \
      "ds_read_b32 v238, v237 offset:576\n" \
      "s_waitcnt lgkmcnt(0)\n" \
      "s_sub_u32 %[sc], %[sc], 1\n" \
      "v_readfirstlane_b32 %[so2], v238\n" \
      "s_cbranch_scc1 LSQ1_%=\n" \
      "s_cmp_lt_u32 %[so2], 0x40000000\n" \
      "s_cbranch_scc0 LSR1_%=\n" \
      "s_nop 3\n" \
      "buffer_load_dwordx4 v[200:203], %[vs0], %[rs], %[so2] offen offset:0 sc0 sc1\n" \
      "buffer_load_dwordx4 v[204:207], %[vs0], %[rs], %[so2] offen offset:1024 sc0 sc1\n" \
      "buffer_load_dwordx4 v[208:211], %[vs1], %[rs], %[so2] offen offset:0 sc0 sc1\n" \
      "buffer_load_dwordx4 v[212:215], %[vs1], %[rs], %[so2] offen offset:1024 sc0 sc1\n" \
      "s_waitcnt vmcnt(4)\n" \
      "ds_read2_b32 v[234:235], v237 offset0:0 offset1:142\n" \
      "v_min3_u32 v232, v184, v188, v192\n" \
      "v_min_u32 v232, v232, v196\n" \
      "v_cmp_eq_u32 vcc, %[xz], v232\n" \
      "s_waitcnt lgkmcnt(0)\n" \
      "s_cbranch_vccz LS1_%=\n" \
      "LSQ1_%=:\n" \
      ".endif\n" \
      "s_memrealtime vcc\n" \
      "s_waitcnt vmcnt(0) lgkmcnt(0)\n" \
      "s_sub_u32 %[sc], vcc_lo, %[t0]\n" \
      "v_min3_u32 v232, v184, v188, v192\n" \
      "v_min_u32 v232, v232, v196\n" \
      "v_cmp_eq_u32 vcc, %[xz], v232\n" \
      "s_cbranch_vccz LSB1_%=\n" \
      "s_sleep %[psl]\n" \
      "s_cmp_lt_u32 %[sc], %[dl]\n" \
      "s_cbranch_scc1 LSP1_%=\n" \
      "s_mov_b32 %[sto], 1\n" \
      "LSB1_%=:\n" \
      "ds_read_b32 v238, v237 offset:576\n" \
      "ds_read2_b32 v[234:235], v237 offset0:0 offset1:142\n" \
      "v_min3_u32 v232, v200, v204, v208\n" \
      "v_min_u32 v232, v232, v212\n" \
      "v_cmp_eq_u32 vcc, %[xz], v232\n" \
      "s_waitcnt lgkmcnt(0)\n" \
      "v_readfirstlane_b32 %[so2], v238\n" \
      "s_or_b64 exec, vcc, 1\n" \
      "s_nop 3\n" \
      "buffer_load_dwordx4 v[200:203], %[vs0], %[rs], %[so2] offen offset:0 sc0 sc1\n" \
      "buffer_load_dwordx4 v[204:207], %[vs0], %[rs], %[so2] offen offset:1024 sc0 sc1\n" \
      "buffer_load_dwordx4 v[208:211], %[vs1], %[rs], %[so2] offen offset:0 sc0 sc1\n" \
      "buffer_load_dwordx4 v[212:215], %[vs1], %[rs], %[so2] offen offset:1024 sc0 sc1\n" \
      "s_mov_b64 exec, -1\n" \
      "s_branch LS1_%=\n" \
      "LSW1_%=:\n" \
      "s_nop 2\n" \
      "v_mfma_f32_16x16x128_f8f6f4 v[44:47], v[112:119], v[184:191], v[44:47] cbsz:1\n" \
      ".if %[xsc] == 0\n" \
      "buffer_load_dwordx4 v[216:219], %[vs0], %[rs], %[so3] offen offset:0 sc0 sc1\n" \
      ".elseif %[xsc] == 1\n" \
      "buffer_load_dwordx4 v[216:219], %[vs0], %[rs], %[so3] offen offset:0\n" \
      ".elseif %[xsc] == 2\n" \
      "buffer_load_dwordx4 v[216:219], %[vs0], %[rs], %[so3] offen offset:0 sc0\n" \
      ".elseif %[xsc] == 3\n" \
      "buffer_load_dwordx4 v[216:219], %[vs0], %[rs], %[so3] offen offset:0 sc1\n" \
      ".else\n" \
      "buffer_load_dwordx4 v[216:219], %[vs0], %[rs], %[so3] offen offset:0 nt\n" \
      ".endif\n" \
      "v_mfma_f32_16x16x128_f8f6f4 v[76:79], v[112:119], v[192:199], v[76:79] cbsz:1\n" \
      "v_mfma_f32_16x16x128_f8f6f4 v[48:51], v[120:127], v[184:191], v[48:51] cbsz:1\n" \
      ".if %[xsc] == 0\n" \
      "buffer_load_dwordx4 v[220:223], %[vs0], %[rs], %[so3] offen offset:1024 sc0 sc1\n" \
      ".elseif %[xsc] == 1\n" \
      "buffer_load_dwordx4 v[220:223], %[vs0], %[rs], %[so3] offen offset:1024\n" \
      ".elseif %[xsc] == 2\n" \
      "buffer_load_dwordx4 v[220:223], %[vs0], %[rs], %[so3] offen offset:1024 sc0\n" \
      ".elseif %[xsc] == 3\n" \
      "buffer_load_dwordx4 v[220:223], %[vs0], %[rs], %[so3] offen offset:1024 sc1\n" \
      ".else\n" \
      "buffer_load_dwordx4 v[220:223], %[vs0], %[rs], %[so3] offen offset:1024 nt\n" \
      ".endif\n" \
      "v_mfma_f32_16x16x128_f8f6f4 v[80:83], v[120:127], v[192:199], v[80:83] cbsz:1\n" \
      "v_mfma_f32_16x16x128_f8f6f4 v[52:55], v[128:135], v[184:191], v[52:55] cbsz:1\n" \
      ".if %[xsc] == 0\n" \
      "buffer_load_dwordx4 v[224:227], %[vs1], %[rs], %[so3] offen offset:0 sc0 sc1\n" \
      ".elseif %[xsc] == 1\n" \
      "buffer_load_dwordx4 v[224:227], %[vs1], %[rs], %[so3] offen offset:0\n" \
      ".elseif %[xsc] == 2\n" \
      "buffer_load_dwordx4 v[224:227], %[vs1], %[rs], %[so3] offen offset:0 sc0\n" \
      ".elseif %[xsc] == 3\n" \
      "buffer_load_dwordx4 v[224:227], %[vs1], %[rs], %[so3] offen offset:0 sc1\n" \
      ".else\n" \
      "buffer_load_dwordx4 v[224:227], %[vs1], %[rs], %[so3] offen offset:0 nt\n" \
      ".endif\n" \
      "v_mfma_f32_16x16x128_f8f6f4 v[84:87], v[128:135], v[192:199], v[84:87] cbsz:1\n" \
      "s_waitcnt lgkmcnt(0)\n" \
      "v_mfma_f32_16x16x128_f8f6f4 v[56:59], v[136:143], v[184:191], v[56:59] cbsz:1\n" \
      ".if %[xsc] == 0\n" \
      "buffer_load_dwordx4 v[228:231], %[vs1], %[rs], %[so3] offen offset:1024 sc0 sc1\n" \
      ".elseif %[xsc] == 1\n" \
      "buffer_load_dwordx4 v[228:231], %[vs1], %[rs], %[so3] offen offset:1024\n" \
      ".elseif %[xsc] == 2\n" \
      "buffer_load_dwordx4 v[228:231], %[vs1], %[rs], %[so3] offen offset:1024 sc0\n" \
      ".elseif %[xsc] == 3\n" \
      "buffer_load_dwordx4 v[228:231], %[vs1], %[rs], %[so3] offen offset:1024 sc1\n" \
      ".else\n" \
      "buffer_load_dwordx4 v[228:231], %[vs1], %[rs], %[so3] offen offset:1024 nt\n" \
      ".endif\n" \
      "v_mfma_f32_16x16x128_f8f6f4 v[88:91], v[136:143], v[192:199], v[88:91] cbsz:1\n" \
      "v_mfma_f32_16x16x128_f8f6f4 v[60:63], v[144:151], v[184:191], v[60:63] cbsz:1\n" \
      "s_add_u32 %[sj], %[sj], 2\n" \
      "v_mfma_f32_16x16x128_f8f6f4 v[92:95], v[144:151], v[192:199], v[92:95] cbsz:1\n" \
      "v_mfma_f32_16x16x128_f8f6f4 v[64:67], v[152:159], v[184:191], v[64:67] cbsz:1\n" \
      "v_mfma_f32_16x16x128_f8f6f4 v[96:99], v[152:159], v[192:199], v[96:99] cbsz:1\n" \
      "v_mfma_f32_16x16x128_f8f6f4 v[68:71], v[160:167], v[184:191], v[68:71] cbsz:1\n" \
      "v_mfma_f32_16x16x128_f8f6f4 v[100:103], v[160:167], v[192:199], v[100:103] cbsz:1\n" \
      "s_mov_b32 %[sn], 0\n" \
      "s_waitcnt vmcnt(4)\n" \
      "v_min3_u32 v232, v200, v204, v208\n" \
      "v_min_u32 v232, v232, v212\n" \
      "v_cmp_eq_u32 vcc, %[xz], v232\n" \
      "s_cbranch_vccz LV1_%=\n" \
      "ds_read_b32 v238, v237 offset:584\n" \
      "s_mov_b32 %[sn], 1\n" \
      "s_waitcnt lgkmcnt(0)\n" \
      "v_readfirstlane_b32 %[so2], v238\n" \
      "s_mov_b64 exec, vcc\n" \
      "s_nop 3\n" \
      "buffer_load_dwordx4 v[200:203], %[vs0], %[rs], %[so2] offen offset:0 sc0 sc1\n" \
      "buffer_load_dwordx4 v[204:207], %[vs0], %[rs], %[so2] offen offset:1024 sc0 sc1\n" \
      "buffer_load_dwordx4 v[208:211], %[vs1], %[rs], %[so2] offen offset:0 sc0 sc1\n" \
      "buffer_load_dwordx4 v[212:215], %[vs1], %[rs], %[so2] offen offset:1024 sc0 sc1\n" \
      "s_mov_b64 exec, -1\n" \
      "LV1_%=:\n" \
      "LW1a_%=:\n" \
      "s_movk_i32 %[sc], 64\n" \
      "LW1b_%=:\n" \
      "ds_read_b32 v234, v237 offset:8\n" \
      "s_waitcnt lgkmcnt(0)\n" \
      "s_sub_u32 %[sc], %[sc], 1\n" \
      "v_readfirstlane_b32 %[sf], v234\n" \
      "s_cmp_lg_u32 %[sf], 0\n" \
      "s_cbranch_scc1 LW1c_%=\n" \
      "s_cmp_lg_u32 %[sc], 0\n" \
      "s_cbranch_scc1 LW1b_%=\n" \
      "s_memrealtime vcc\n" \
      "s_waitcnt lgkmcnt(0)\n" \
      "s_sub_u32 %[sc], vcc_lo, %[t0]\n" \
      "s_cmp_lt_u32 %[sc], %[dl]\n" \
      "s_cbranch_scc1 LW1a_%=\n" \
      "s_mov_b32 %[sto], 1\n" \
      "LW1c_%=:\n" \
      "ds_read_b64_tr_b8 v[104:105], v32 offset:0\n" \
      "ds_read_b64_tr_b8 v[106:107], v32 offset:1024\n" \
      "ds_read_b64_tr_b8 v[108:109], v32 offset:8192\n" \
      "ds_read_b64_tr_b8 v[110:111], v32 offset:9216\n" \
      "ds_read_b64_tr_b8 v[112:113], v33 offset:0\n" \
      "ds_read_b64_tr_b8 v[114:115], v33 offset:1024\n" \
      "ds_read_b64_tr_b8 v[116:117], v33 offset:8192\n" \
      "ds_read_b64_tr_b8 v[118:119], v33 offset:9216\n" \
      "ds_read_b64_tr_b8 v[120:121], v34 offset:0\n" \
      "ds_read_b64_tr_b8 v[122:123], v34 offset:1024\n" \
      "ds_read_b64_tr_b8 v[124:125], v34 offset:8192\n" \
      "ds_read_b64_tr_b8 v[126:127], v34 offset:9216\n" \
      "ds_read_b64_tr_b8 v[128:129], v35 offset:0\n" \
      "ds_read_b64_tr_b8 v[130:131], v35 offset:1024\n" \
      "ds_read_b64_tr_b8 v[132:133], v35 offset:8192\n" \
      "ds_read_b64_tr_b8 v[134:135], v35 offset:9216\n" \
      "ds_read2_b32 v[234:235], v237 offset0:0 offset1:142\n" \
      "ds_read_b64_tr_b8 v[136:137], v36 offset:0\n" \
      "ds_read_b64_tr_b8 v[138:139], v36 offset:1024\n" \
      "ds_read_b64_tr_b8 v[140:141], v36 offset:8192\n" \
      "ds_read_b64_tr_b8 v[142:143], v36 offset:9216\n" \
      "ds_read_b64_tr_b8 v[144:145], v37 offset:0\n" \
      "ds_read_b64_tr_b8 v[146:147], v37 offset:1024\n" \
      "ds_read_b64_tr_b8 v[148:149], v37 offset:8192\n" \
      "ds_read_b64_tr_b8 v[150:151], v37 offset:9216\n" \
      "ds_read_b64_tr_b8 v[152:153], v38 offset:0\n" \
      "ds_read_b64_tr_b8 v[154:155], v38 offset:1024\n" \
      "ds_read_b64_tr_b8 v[156:157], v38 offset:8192\n" \
      "ds_read_b64_tr_b8 v[158:159], v38 offset:9216\n" \
      "ds_read_b64_tr_b8 v[160:161], v39 offset:0\n" \
      "ds_read_b64_tr_b8 v[162:163], v39 offset:1024\n" \
      "s_cmp_eq_u32 %[sn], 0\n" \
      "s_cbranch_scc1 LS2_%=\n" \
      "s_waitcnt vmcnt(0)\n" \
      "v_min3_u32 v232, v200, v204, v208\n" \
      "v_min_u32 v232, v232, v212\n" \
      "v_cmp_eq_u32 vcc, %[xz], v232\n" \
      "s_cbranch_vccz LSB2_%=\n" \
      "LSA2_%=:\n" \
      "s_cmp_lt_u32 %[so2], 0x40000000\n" \
      "s_cbranch_scc1 LSP2_%=\n" \
      "LO2a_%=:\n" \
      "s_movk_i32 %[sc], 64\n" \
      "LO2b_%=:\n" \
      "ds_read_b32 v238, v237 offset:584\n" \
      "s_waitcnt lgkmcnt(0)\n" \
      "s_sub_u32 %[sc], %[sc], 1\n" \
      "v_readfirstlane_b32 %[sf], v238\n" \
      "s_cmp_lt_u32 %[sf], 0x40000000\n" \
      "s_cbranch_scc1 LO2c_%=\n" \
      "s_cmp_lg_u32 %[sc], 0\n" \
      "s_cbranch_scc1 LO2b_%=\n" \
      "s_memrealtime vcc\n" \
      "s_waitcnt lgkmcnt(0)\n" \
      "s_sub_u32 %[sc], vcc_lo, %[t0]\n" \
      "s_cmp_lt_u32 %[sc], %[dl]\n" \
      "s_cbranch_scc1 LO2a_%=\n" \
      "s_mov_b32 %[sto], 1\n" \
      "LO2c_%=:\n" \
      "s_mov_b32 %[so2], %[sf]\n" \
      "s_mov_b64 vcc, -1\n" \
      "LSP2_%=:\n" \
      "s_mov_b64 exec, vcc\n" \
      "buffer_load_dwordx4 v[200:203], %[vs0], %[rs], %[so2] offen offset:0 sc0 sc1\n" \
      "buffer_load_dwordx4 v[204:207], %[vs0], %[rs], %[so2] offen offset:1024 sc0 sc1\n" \
      "buffer_load_dwordx4 v[208:211], %[vs1], %[rs], %[so2] offen offset:0 sc0 sc1\n" \
      "buffer_load_dwordx4 v[212:215], %[vs1], %[rs], %[so2] offen offset:1024 sc0 sc1\n" \
      "s_mov_b64 exec, -1\n" \
      ".if %[npl]\n" \
      "s_cmp_lt_u32 %[so3], 0x40000000\n" \
      "s_cbranch_scc1 LSQ2_%=\n" \
      "s_movk_i32 %[sc], %[npl]\n" \
      "LSR2_%=:\n" \
      "ds_read_b32 v238, v237 offset:576\n" \
      "s_waitcnt lgkmcnt(0)\n" \
      "s_sub_u32 %[sc], %[sc], 1\n" \
      "v_readfirstlane_b32 %[so3], v238\n" \
      "s_cbranch_scc1 LSQ2_%=\n" \
      "s_cmp_lt_u32 %[so3], 0x40000000\n" \
      "s_cbranch_scc0 LSR2_%=\n" \
      "s_nop 3\n" \
      "buffer_load_dwordx4 v[216:219], %[vs0], %[rs], %[so3] offen offset:0 sc0 sc1\n" \
      "buffer_load_dwordx4 v[220:223], %[vs0], %[rs], %[so3] offen offset:1024 sc0 sc1\n" \
      "buffer_load_dwordx4 v[224:227], %[vs1], %[rs], %[so3] offen offset:0 sc0 sc1\n" \
      "buffer_load_dwordx4 v[228:231], %[vs1], %[rs], %[so3] offen offset:1024 sc0 sc1\n" \
      "s_waitcnt vmcnt(4)\n" \
      "ds_read2_b32 v[234:235], v237 offset0:0 offset1:142\n" \
      "v_min3_u32 v232, v200, v204, v208\n" \
      "v_min_u32 v232, v232, v212\n" \
      "v_cmp_eq_u32 vcc, %[xz], v232\n" \
      "s_waitcnt lgkmcnt(0)\n" \
      "s_cbranch_vccz LS2_%=\n" \
      "LSQ2_%=:\n" \
      ".endif\n" \
      "s_memrealtime vcc\n" \
      "s_waitcnt vmcnt(0) lgkmcnt(0)\n" \
      "s_sub_u32 %[sc], vcc_lo, %[t0]\n" \
      "v_min3_u32 v232, v200, v204, v208\n" \
      "v_min_u32 v232, v232, v212\n" \
      "v_cmp_eq_u32 vcc, %[xz], v232\n" \
      "s_cbranch_vccz LSB2_%=\n" \
      "s_sleep %[psl]\n" \
      "s_cmp_lt_u32 %[sc], %[dl]\n" \
      "s_cbranch_scc1 LSP2_%=\n" \
      "s_mov_b32 %[sto], 1\n" \
      "LSB2_%=:\n" \
      "ds_read_b32 v238, v237 offset:576\n" \
      "ds_read2_b32 v[234:235], v237 offset0:0 offset1:142\n" \
      "v_min3_u32 v232, v216, v220, v224\n" \
      "v_min_u32 v232, v232, v228\n" \
      "v_cmp_eq_u32 vcc, %[xz], v232\n" \
      "s_waitcnt lgkmcnt(0)\n" \
      "v_readfirstlane_b32 %[so3], v238\n" \
      "s_or_b64 exec, vcc, 1\n" \
      "s_nop 3\n" \
      "buffer_load_dwordx4 v[216:219], %[vs0], %[rs], %[so3] offen offset:0 sc0 sc1\n" \
      "buffer_load_dwordx4 v[220:223], %[vs0], %[rs], %[so3] offen offset:1024 sc0 sc1\n" \
      "buffer_load_dwordx4 v[224:227], %[vs1], %[rs], %[so3] offen offset:0 sc0 sc1\n" \
      "buffer_load_dwordx4 v[228:231], %[vs1], %[rs], %[so3] offen offset:1024 sc0 sc1\n" \
      "s_mov_b64 exec, -1\n" \
      "s_branch LS2_%=\n" \
      "LSW2_%=:\n" \
      "s_nop 2\n" \
      "v_mfma_f32_16x16x128_f8f6f4 v[44:47], v[112:119], v[200:207], v[44:47] cbsz:1\n" \
      ".if %[xsc] == 0\n" \
      "buffer_load_dwordx4 v[168:171], %[vs0], %[rs], %[so0] offen offset:0 sc0 sc1\n" \
      ".elseif %[xsc] == 1\n" \
      "buffer_load_dwordx4 v[168:171], %[vs0], %[rs], %[so0] offen offset:0\n" \
      ".elseif %[xsc] == 2\n" \
      "buffer_load_dwordx4 v[168:171], %[vs0], %[rs], %[so0] offen offset:0 sc0\n" \
      ".elseif %[xsc] == 3\n" \
      "buffer_load_dwordx4 v[168:171], %[vs0], %[rs], %[so0] offen offset:0 sc1\n" \
      ".else\n" \
      "buffer_load_dwordx4 v[168:171], %[vs0], %[rs], %[so0] offen offset:0 nt\n" \
      ".endif\n" \
      "v_mfma_f32_16x16x128_f8f6f4 v[76:79], v[112:119], v[208:215], v[76:79] cbsz:1\n" \
      "v_mfma_f32_16x16x128_f8f6f4 v[48:51], v[120:127], v[200:207], v[48:51] cbsz:1\n" \
      ".if %[xsc] == 0\n" \
      "buffer_load_dwordx4 v[172:175], %[vs0], %[rs], %[so0] offen offset:1024 sc0 sc1\n" \
      ".elseif %[xsc] == 1\n" \
      "buffer_load_dwordx4 v[172:175], %[vs0], %[rs], %[so0] offen offset:1024\n" \
      ".elseif %[xsc] == 2\n" \
      "buffer_load_dwordx4 v[172:175], %[vs0], %[rs], %[so0] offen offset:1024 sc0\n" \
      ".elseif %[xsc] == 3\n" \
      "buffer_load_dwordx4 v[172:175], %[vs0], %[rs], %[so0] offen offset:1024 sc1\n" \
      ".else\n" \
      "buffer_load_dwordx4 v[172:175], %[vs0], %[rs], %[so0] offen offset:1024 nt\n" \
      ".endif\n" \
      "v_mfma_f32_16x16x128_f8f6f4 v[80:83], v[120:127], v[208:215], v[80:83] cbsz:1\n" \
      "v_mfma_f32_16x16x128_f8f6f4 v[52:55], v[128:135], v[200:207], v[52:55] cbsz:1\n" \
      ".if %[xsc] == 0\n" \
      "buffer_load_dwordx4 v[176:179], %[vs1], %[rs], %[so0] offen offset:0 sc0 sc1\n" \
      ".elseif %[xsc] == 1\n" \
      "buffer_load_dwordx4 v[176:179], %[vs1], %[rs], %[so0] offen offset:0\n" \
      ".elseif %[xsc] == 2\n" \
      "buffer_load_dwordx4 v[176:179], %[vs1], %[rs], %[so0] offen offset:0 sc0\n" \
      ".elseif %[xsc] == 3\n" \
      "buffer_load_dwordx4 v[176:179], %[vs1], %[rs], %[so0] offen offset:0 sc1\n" \
      ".else\n" \
      "buffer_load_dwordx4 v[176:179], %[vs1], %[rs], %[so0] offen offset:0 nt\n" \
      ".endif\n" \
      "v_mfma_f32_16x16x128_f8f6f4 v[84:87], v[128:135], v[208:215], v[84:87] cbsz:1\n" \
      "s_waitcnt lgkmcnt(0)\n" \
      "v_mfma_f32_16x16x128_f8f6f4 v[56:59], v[136:143], v[200:207], v[56:59] cbsz:1\n" \
      ".if %[xsc] == 0\n" \
      "buffer_load_dwordx4 v[180:183], %[vs1], %[rs], %[so0] offen offset:1024 sc0 sc1\n" \
      ".elseif %[xsc] == 1\n" \
      "buffer_load_dwordx4 v[180:183], %[vs1], %[rs], %[so0] offen offset:1024\n" \
      ".elseif %[xsc] == 2\n" \
      "buffer_load_dwordx4 v[180:183], %[vs1], %[rs], %[so0] offen offset:1024 sc0\n" \
      ".elseif %[xsc] == 3\n" \
      "buffer_load_dwordx4 v[180:183], %[vs1], %[rs], %[so0] offen offset:1024 sc1\n" \
      ".else\n" \
      "buffer_load_dwordx4 v[180:183], %[vs1], %[rs], %[so0] offen offset:1024 nt\n" \
      ".endif\n" \
      "v_mfma_f32_16x16x128_f8f6f4 v[88:91], v[136:143], v[208:215], v[88:91] cbsz:1\n" \
      "v_mfma_f32_16x16x128_f8f6f4 v[60:63], v[144:151], v[200:207], v[60:63] cbsz:1\n" \
      "s_add_u32 %[sj], %[sj], 2\n" \
      "v_mfma_f32_16x16x128_f8f6f4 v[92:95], v[144:151], v[208:215], v[92:95] cbsz:1\n" \
      "v_mfma_f32_16x16x128_f8f6f4 v[64:67], v[152:159], v[200:207], v[64:67] cbsz:1\n" \
      "v_mfma_f32_16x16x128_f8f6f4 v[96:99], v[152:159], v[208:215], v[96:99] cbsz:1\n" \
      "v_mfma_f32_16x16x128_f8f6f4 v[68:71], v[160:167], v[200:207], v[68:71] cbsz:1\n" \
      "v_mfma_f32_16x16x128_f8f6f4 v[100:103], v[160:167], v[208:215], v[100:103] cbsz:1\n" \
      "s_mov_b32 %[sn], 0\n" \
      "s_waitcnt vmcnt(4)\n" \
      "v_min3_u32 v232, v216, v220, v224\n" \
      "v_min_u32 v232, v232, v228\n" \
      "v_cmp_eq_u32 vcc, %[xz], v232\n" \
      "s_cbranch_vccz LV2_%=\n" \
      "ds_read_b32 v238, v237 offset:584\n" \
      "s_mov_b32 %[sn], 1\n" \
      "s_waitcnt lgkmcnt(0)\n" \
      "v_readfirstlane_b32 %[so3], v238\n" \
      "s_mov_b64 exec, vcc\n" \
      "s_nop 3\n" \
      "buffer_load_dwordx4 v[216:219], %[vs0], %[rs], %[so3] offen offset:0 sc0 sc1\n" \
      "buffer_load_dwordx4 v[220:223], %[vs0], %[rs], %[so3] offen offset:1024 sc0 sc1\n" \
      "buffer_load_dwordx4 v[224:227], %[vs1], %[rs], %[so3] offen offset:0 sc0 sc1\n" \
      "buffer_load_dwordx4 v[228:231], %[vs1], %[rs], %[so3] offen offset:1024 sc0 sc1\n" \
      "s_mov_b64 exec, -1\n" \
      "LV2_%=:\n" \
      "LW2a_%=:\n" \
      "s_movk_i32 %[sc], 64\n" \
      "LW2b_%=:\n" \
      "ds_read_b32 v234, v237 offset:8\n" \
      "s_waitcnt lgkmcnt(0)\n" \
      "s_sub_u32 %[sc], %[sc], 1\n" \
      "v_readfirstlane_b32 %[sf], v234\n" \
      "s_cmp_lg_u32 %[sf], 0\n" \
      "s_cbranch_scc1 LW2c_%=\n" \
      "s_cmp_lg_u32 %[sc], 0\n" \
      "s_cbranch_scc1 LW2b_%=\n" \
      "s_memrealtime vcc\n" \
      "s_waitcnt lgkmcnt(0)\n" \
      "s_sub_u32 %[sc], vcc_lo, %[t0]\n" \
      "s_cmp_lt_u32 %[sc], %[dl]\n" \
      "s_cbranch_scc1 LW2a_%=\n" \
      "s_mov_b32 %[sto], 1\n" \
      "LW2c_%=:\n" \
      "ds_read_b64_tr_b8 v[104:105], v32 offset:32768\n" \
      "ds_read_b64_tr_b8 v[106:107], v32 offset:33792\n" \
      "ds_read_b64_tr_b8 v[108:109], v32 offset:40960\n" \
      "ds_read_b64_tr_b8 v[110:111], v32 offset:41984\n" \
      "ds_read_b64_tr_b8 v[112:113], v33 offset:32768\n" \
      "ds_read_b64_tr_b8 v[114:115], v33 offset:33792\n" \
      "ds_read_b64_tr_b8 v[116:117], v33 offset:40960\n" \
      "ds_read_b64_tr_b8 v[118:119], v33 offset:41984\n" \
      "ds_read_b64_tr_b8 v[120:121], v34 offset:32768\n" \
      "ds_read_b64_tr_b8 v[122:123], v34 offset:33792\n" \
      "ds_read_b64_tr_b8 v[124:125], v34 offset:40960\n" \
      "ds_read_b64_tr_b8 v[126:127], v34 offset:41984\n" \
      "ds_read_b64_tr_b8 v[128:129], v35 offset:32768\n" \
      "ds_read_b64_tr_b8 v[130:131], v35 offset:33792\n" \
      "ds_read_b64_tr_b8 v[132:133], v35 offset:40960\n" \
      "ds_read_b64_tr_b8 v[134:135], v35 offset:41984\n" \
      "ds_read2_b32 v[234:235], v237 offset0:0 offset1:142\n" \
      "ds_read_b64_tr_b8 v[136:137], v36 offset:32768\n" \
      "ds_read_b64_tr_b8 v[138:139], v36 offset:33792\n" \
      "ds_read_b64_tr_b8 v[140:141], v36 offset:40960\n" \
      "ds_read_b64_tr_b8 v[142:143], v36 offset:41984\n" \
      "ds_read_b64_tr_b8 v[144:145], v37 offset:32768\n" \
      "ds_read_b64_tr_b8 v[146:147], v37 offset:33792\n" \
      "ds_read_b64_tr_b8 v[148:149], v37 offset:40960\n" \
      "ds_read_b64_tr_b8 v[150:151], v37 offset:41984\n" \
      "ds_read_b64_tr_b8 v[152:153], v38 offset:32768\n" \
      "ds_read_b64_tr_b8 v[154:155], v38 offset:33792\n" \
      "ds_read_b64_tr_b8 v[156:157], v38 offset:40960\n" \
      "ds_read_b64_tr_b8 v[158:159], v38 offset:41984\n" \
      "ds_read_b64_tr_b8 v[160:161], v39 offset:32768\n" \
      "ds_read_b64_tr_b8 v[162:163], v39 offset:33792\n" \
      "s_cmp_eq_u32 %[sn], 0\n" \
      "s_cbranch_scc1 LS3_%=\n" \
      "s_waitcnt vmcnt(0)\n" \
      "v_min3_u32 v232, v216, v220, v224\n" \
      "v_min_u32 v232, v232, v228\n" \
      "v_cmp_eq_u32 vcc, %[xz], v232\n" \
      "s_cbranch_vccz LSB3_%=\n" \
      "LSA3_%=:\n" \
      "s_cmp_lt_u32 %[so3], 0x40000000\n" \
      "s_cbranch_scc1 LSP3_%=\n" \
      "LO3a_%=:\n" \
      "s_movk_i32 %[sc], 64\n" \
      "LO3b_%=:\n" \
      "ds_read_b32 v238, v237 offset:584\n" \
      "s_waitcnt lgkmcnt(0)\n" \
      "s_sub_u32 %[sc], %[sc], 1\n" \
      "v_readfirstlane_b32 %[sf], v238\n" \
      "s_cmp_lt_u32 %[sf], 0x40000000\n" \
      "s_cbranch_scc1 LO3c_%=\n" \
      "s_cmp_lg_u32 %[sc], 0\n" \
      "s_cbranch_scc1 LO3b_%=\n" \
      "s_memrealtime vcc\n" \
      "s_waitcnt lgkmcnt(0)\n" \
      "s_sub_u32 %[sc], vcc_lo, %[t0]\n" \
      "s_cmp_lt_u32 %[sc], %[dl]\n" \
      "s_cbranch_scc1 LO3a_%=\n" \
      "s_mov_b32 %[sto], 1\n" \
      "LO3c_%=:\n" \
      "s_mov_b32 %[so3], %[sf]\n" \
      "s_mov_b64 vcc, -1\n" \
      "LSP3_%=:\n" \
      "s_mov_b64 exec, vcc\n" \
      "buffer_load_dwordx4 v[216:219], %[vs0], %[rs], %[so3] offen offset:0 sc0 sc1\n" \
      "buffer_load_dwordx4 v[220:223], %[vs0], %[rs], %[so3] offen offset:1024 sc0 sc1\n" \
      "buffer_load_dwordx4 v[224:227], %[vs1], %[rs], %[so3] offen offset:0 sc0 sc1\n" \
      "buffer_load_dwordx4 v[228:231], %[vs1], %[rs], %[so3] offen offset:1024 sc0 sc1\n" \
      "s_mov_b64 exec, -1\n" \
      ".if %[npl]\n" \
      "s_cmp_lt_u32 %[so0], 0x40000000\n" \
      "s_cbranch_scc1 LSQ3_%=\n" \
      "s_movk_i32 %[sc], %[npl]\n" \
      "LSR3_%=:\n" \
      "ds_read_b32 v238, v237 offset:576\n" \
      "s_waitcnt lgkmcnt(0)\n" \
      "s_sub_u32 %[sc], %[sc], 1\n" \
      "v_readfirstlane_b32 %[so0], v238\n" \
      "s_cbranch_scc1 LSQ3_%=\n" \
      "s_cmp_lt_u32 %[so0], 0x40000000\n" \
      "s_cbranch_scc0 LSR3_%=\n" \
      "s_nop 3\n" \
      "buffer_load_dwordx4 v[168:171], %[vs0], %[rs], %[so0] offen offset:0 sc0 sc1\n" \
      "buffer_load_dwordx4 v[172:175], %[vs0], %[rs], %[so0] offen offset:1024 sc0 sc1\n" \
      "buffer_load_dwordx4 v[176:179], %[vs1], %[rs], %[so0] offen offset:0 sc0 sc1\n" \
      "buffer_load_dwordx4 v[180:183], %[vs1], %[rs], %[so0] offen offset:1024 sc0 sc1\n" \
      "s_waitcnt vmcnt(4)\n" \
      "ds_read2_b32 v[234:235], v237 offset0:0 offset1:142\n" \
      "v_min3_u32 v232, v216, v220, v224\n" \
      "v_min_u32 v232, v232, v228\n" \
      "v_cmp_eq_u32 vcc, %[xz], v232\n" \
      "s_waitcnt lgkmcnt(0)\n" \
      "s_cbranch_vccz LS3_%=\n" \
      "LSQ3_%=:\n" \
      ".endif\n" \
      "s_memrealtime vcc\n" \
      "s_waitcnt vmcnt(0) lgkmcnt(0)\n" \
      "s_sub_u32 %[sc], vcc_lo, %[t0]\n" \
      "v_min3_u32 v232, v216, v220, v224\n" \
      "v_min_u32 v232, v232, v228\n" \
      "v_cmp_eq_u32 vcc, %[xz], v232\n" \
      "s_cbranch_vccz LSB3_%=\n" \
      "s_sleep %[psl]\n" \
      "s_cmp_lt_u32 %[sc], %[dl]\n" \
      "s_cbranch_scc1 LSP3_%=\n" \
      "s_mov_b32 %[sto], 1\n" \
      "LSB3_%=:\n" \
      "ds_read_b32 v238, v237 offset:576\n" \
      "ds_read2_b32 v[234:235], v237 offset0:0 offset1:142\n" \
      "v_min3_u32 v232, v168, v172, v176\n" \
      "v_min_u32 v232, v232, v180\n" \
      "v_cmp_eq_u32 vcc, %[xz], v232\n" \
      "s_waitcnt lgkmcnt(0)\n" \
      "v_readfirstlane_b32 %[so0], v238\n" \
      "s_or_b64 exec, vcc, 1\n" \
      "s_nop 3\n" \
      "buffer_load_dwordx4 v[168:171], %[vs0], %[rs], %[so0] offen offset:0 sc0 sc1\n" \
      "buffer_load_dwordx4 v[172:175], %[vs0], %[rs], %[so0] offen offset:1024 sc0 sc1\n" \
      "buffer_load_dwordx4 v[176:179], %[vs1], %[rs], %[so0] offen offset:0 sc0 sc1\n" \
      "buffer_load_dwordx4 v[180:183], %[vs1], %[rs], %[so0] offen offset:1024 sc0 sc1\n" \
      "s_mov_b64 exec, -1\n" \
      "s_branch LS3_%=\n" \
      "LSW3_%=:\n" \
      "s_nop 2\n" \
      "v_mfma_f32_16x16x128_f8f6f4 v[44:47], v[112:119], v[216:223], v[44:47] cbsz:1\n" \
      ".if %[xsc] == 0\n" \
      "buffer_load_dwordx4 v[184:187], %[vs0], %[rs], %[so1] offen offset:0 sc0 sc1\n" \
      ".elseif %[xsc] == 1\n" \
      "buffer_load_dwordx4 v[184:187], %[vs0], %[rs], %[so1] offen offset:0\n" \
      ".elseif %[xsc] == 2\n" \
      "buffer_load_dwordx4 v[184:187], %[vs0], %[rs], %[so1] offen offset:0 sc0\n" \
      ".elseif %[xsc] == 3\n" \
      "buffer_load_dwordx4 v[184:187], %[vs0], %[rs], %[so1] offen offset:0 sc1\n" \
      ".else\n" \
      "buffer_load_dwordx4 v[184:187], %[vs0], %[rs], %[so1] offen offset:0 nt\n" \
      ".endif\n" \
      "v_mfma_f32_16x16x128_f8f6f4 v[76:79], v[112:119], v[224:231], v[76:79] cbsz:1\n" \
      "v_mfma_f32_16x16x128_f8f6f4 v[48:51], v[120:127], v[216:223], v[48:51] cbsz:1\n" \
      ".if %[xsc] == 0\n" \
      "buffer_load_dwordx4 v[188:191], %[vs0], %[rs], %[so1] offen offset:1024 sc0 sc1\n" \
      ".elseif %[xsc] == 1\n" \
      "buffer_load_dwordx4 v[188:191], %[vs0], %[rs], %[so1] offen offset:1024\n" \
      ".elseif %[xsc] == 2\n" \
      "buffer_load_dwordx4 v[188:191], %[vs0], %[rs], %[so1] offen offset:1024 sc0\n" \
      ".elseif %[xsc] == 3\n" \
      "buffer_load_dwordx4 v[188:191], %[vs0], %[rs], %[so1] offen offset:1024 sc1\n" \
      ".else\n" \
      "buffer_load_dwordx4 v[188:191], %[vs0], %[rs], %[so1] offen offset:1024 nt\n" \
      ".endif\n" \
      "v_mfma_f32_16x16x128_f8f6f4 v[80:83], v[120:127], v[224:231], v[80:83] cbsz:1\n" \
      "v_mfma_f32_16x16x128_f8f6f4 v[52:55], v[128:135], v[216:223], v[52:55] cbsz:1\n" \
      ".if %[xsc] == 0\n" \
      "buffer_load_dwordx4 v[192:195], %[vs1], %[rs], %[so1] offen offset:0 sc0 sc1\n" \
      ".elseif %[xsc] == 1\n" \
      "buffer_load_dwordx4 v[192:195], %[vs1], %[rs], %[so1] offen offset:0\n" \
      ".elseif %[xsc] == 2\n" \
      "buffer_load_dwordx4 v[192:195], %[vs1], %[rs], %[so1] offen offset:0 sc0\n" \
      ".elseif %[xsc] == 3\n" \
      "buffer_load_dwordx4 v[192:195], %[vs1], %[rs], %[so1] offen offset:0 sc1\n" \
      ".else\n" \
      "buffer_load_dwordx4 v[192:195], %[vs1], %[rs], %[so1] offen offset:0 nt\n" \
      ".endif\n" \
      "v_mfma_f32_16x16x128_f8f6f4 v[84:87], v[128:135], v[224:231], v[84:87] cbsz:1\n" \
      "s_waitcnt lgkmcnt(0)\n" \
      "v_mfma_f32_16x16x128_f8f6f4 v[56:59], v[136:143], v[216:223], v[56:59] cbsz:1\n" \
      ".if %[xsc] == 0\n" \
      "buffer_load_dwordx4 v[196:199], %[vs1], %[rs], %[so1] offen offset:1024 sc0 sc1\n" \
      ".elseif %[xsc] == 1\n" \
      "buffer_load_dwordx4 v[196:199], %[vs1], %[rs], %[so1] offen offset:1024\n" \
      ".elseif %[xsc] == 2\n" \
      "buffer_load_dwordx4 v[196:199], %[vs1], %[rs], %[so1] offen offset:1024 sc0\n" \
      ".elseif %[xsc] == 3\n" \
      "buffer_load_dwordx4 v[196:199], %[vs1], %[rs], %[so1] offen offset:1024 sc1\n" \
      ".else\n" \
      "buffer_load_dwordx4 v[196:199], %[vs1], %[rs], %[so1] offen offset:1024 nt\n" \
      ".endif\n" \
      "v_mfma_f32_16x16x128_f8f6f4 v[88:91], v[136:143], v[224:231], v[88:91] cbsz:1\n" \
      "v_mfma_f32_16x16x128_f8f6f4 v[60:63], v[144:151], v[216:223], v[60:63] cbsz:1\n" \
      "s_add_u32 %[sj], %[sj], 2\n" \
      "v_mfma_f32_16x16x128_f8f6f4 v[92:95], v[144:151], v[224:231], v[92:95] cbsz:1\n" \
      "v_mfma_f32_16x16x128_f8f6f4 v[64:67], v[152:159], v[216:223], v[64:67] cbsz:1\n" \
      "v_mfma_f32_16x16x128_f8f6f4 v[96:99], v[152:159], v[224:231], v[96:99] cbsz:1\n" \
      "v_mfma_f32_16x16x128_f8f6f4 v[68:71], v[160:167], v[216:223], v[68:71] cbsz:1\n" \
      "v_mfma_f32_16x16x128_f8f6f4 v[100:103], v[160:167], v[224:231], v[100:103] cbsz:1\n" \
      "s_mov_b32 %[sn], 0\n" \
      "s_waitcnt vmcnt(4)\n" \
      "v_min3_u32 v232, v168, v172, v176\n" \
      "v_min_u32 v232, v232, v180\n" \
      "v_cmp_eq_u32 vcc, %[xz], v232\n" \
      "s_cbranch_vccz LV3_%=\n" \
      "s_cmp_lt_u32 %[sj], 4\n" \
      "s_cbranch_scc1 LV3_%=\n" \
      "ds_read_b32 v238, v237 offset:584\n" \
      "s_mov_b32 %[sn], 1\n" \
      "s_waitcnt lgkmcnt(0)\n" \
      "v_readfirstlane_b32 %[so0], v238\n" \
      "s_mov_b64 exec, vcc\n" \
      "s_nop 3\n" \
      "buffer_load_dwordx4 v[168:171], %[vs0], %[rs], %[so0] offen offset:0 sc0 sc1\n" \
      "buffer_load_dwordx4 v[172:175], %[vs0], %[rs], %[so0] offen offset:1024 sc0 sc1\n" \
      "buffer_load_dwordx4 v[176:179], %[vs1], %[rs], %[so0] offen offset:0 sc0 sc1\n" \
      "buffer_load_dwordx4 v[180:183], %[vs1], %[rs], %[so0] offen offset:1024 sc0 sc1\n" \
      "s_mov_b64 exec, -1\n" \
      "LV3_%=:\n" \
      "LW3a_%=:\n" \
      "s_movk_i32 %[sc], 64\n" \
      "LW3b_%=:\n" \
      "ds_read_b32 v234, v237 offset:8\n" \
      "s_waitcnt lgkmcnt(0)\n" \
      "s_sub_u32 %[sc], %[sc], 1\n" \
      "v_readfirstlane_b32 %[sf], v234\n" \
      "s_cmp_lg_u32 %[sf], 0\n" \
      "s_cbranch_scc1 LW3c_%=\n" \
      "s_cmp_lg_u32 %[sc], 0\n" \
      "s_cbranch_scc1 LW3b_%=\n" \
      "s_memrealtime vcc\n" \
      "s_waitcnt lgkmcnt(0)\n" \
      "s_sub_u32 %[sc], vcc_lo, %[t0]\n" \
      "s_cmp_lt_u32 %[sc], %[dl]\n" \
      "s_cbranch_scc1 LW3a_%=\n" \
      "s_mov_b32 %[sto], 1\n" \
      "LW3c_%=:\n" \
      "ds_read_b64_tr_b8 v[104:105], v24 offset:0\n" \
      "ds_read_b64_tr_b8 v[106:107], v24 offset:1024\n" \
      "ds_read_b64_tr_b8 v[108:109], v24 offset:8192\n" \
      "ds_read_b64_tr_b8 v[110:111], v24 offset:9216\n" \
      "ds_read_b64_tr_b8 v[112:113], v25 offset:0\n" \
      "ds_read_b64_tr_b8 v[114:115], v25 offset:1024\n" \
      "ds_read_b64_tr_b8 v[116:117], v25 offset:8192\n" \
      "ds_read_b64_tr_b8 v[118:119], v25 offset:9216\n" \
      "ds_read_b64_tr_b8 v[120:121], v26 offset:0\n" \
      "ds_read_b64_tr_b8 v[122:123], v26 offset:1024\n" \
      "ds_read_b64_tr_b8 v[124:125], v26 offset:8192\n" \
      "ds_read_b64_tr_b8 v[126:127], v26 offset:9216\n" \
      "ds_read_b64_tr_b8 v[128:129], v27 offset:0\n" \
      "ds_read_b64_tr_b8 v[130:131], v27 offset:1024\n" \
      "ds_read_b64_tr_b8 v[132:133], v27 offset:8192\n" \
      "ds_read_b64_tr_b8 v[134:135], v27 offset:9216\n" \
      "ds_read2_b32 v[234:235], v237 offset0:0 offset1:142\n" \
      "ds_read_b64_tr_b8 v[136:137], v28 offset:0\n" \
      "ds_read_b64_tr_b8 v[138:139], v28 offset:1024\n" \
      "ds_read_b64_tr_b8 v[140:141], v28 offset:8192\n" \
      "ds_read_b64_tr_b8 v[142:143], v28 offset:9216\n" \
      "ds_read_b64_tr_b8 v[144:145], v29 offset:0\n" \
      "ds_read_b64_tr_b8 v[146:147], v29 offset:1024\n" \
      "ds_read_b64_tr_b8 v[148:149], v29 offset:8192\n" \
      "ds_read_b64_tr_b8 v[150:151], v29 offset:9216\n" \
      "ds_read_b64_tr_b8 v[152:153], v30 offset:0\n" \
      "ds_read_b64_tr_b8 v[154:155], v30 offset:1024\n" \
      "ds_read_b64_tr_b8 v[156:157], v30 offset:8192\n" \
      "ds_read_b64_tr_b8 v[158:159], v30 offset:9216\n" \
      "ds_read_b64_tr_b8 v[160:161], v31 offset:0\n" \
      "ds_read_b64_tr_b8 v[162:163], v31 offset:1024\n" \
      "s_cmp_eq_u32 %[sn], 0\n" \
      "s_cbranch_scc1 LS0_%=\n" \
      "s_waitcnt vmcnt(0)\n" \
      "v_min3_u32 v232, v168, v172, v176\n" \
      "v_min_u32 v232, v232, v180\n" \
      "v_cmp_eq_u32 vcc, %[xz], v232\n" \
      "s_cbranch_vccz LSB0_%=\n" \
      "LSA0_%=:\n" \
      "s_cmp_lt_u32 %[sj], 4\n" \
      "s_cbranch_scc1 LS0_%=\n" \
      "s_cmp_lt_u32 %[so0], 0x40000000\n" \
      "s_cbranch_scc1 LSP0_%=\n" \
      "LO0a_%=:\n" \
      "s_movk_i32 %[sc], 64\n" \
      "LO0b_%=:\n" \
      "ds_read_b32 v238, v237 offset:584\n" \
      "s_waitcnt lgkmcnt(0)\n" \
      "s_sub_u32 %[sc], %[sc], 1\n" \
      "v_readfirstlane_b32 %[sf], v238\n" \
      "s_cmp_lt_u32 %[sf], 0x40000000\n" \
      "s_cbranch_scc1 LO0c_%=\n" \
      "s_cmp_lg_u32 %[sc], 0\n" \
      "s_cbranch_scc1 LO0b_%=\n" \
      "s_memrealtime vcc\n" \
      "s_waitcnt lgkmcnt(0)\n" \
      "s_sub_u32 %[sc], vcc_lo, %[t0]\n" \
      "s_cmp_lt_u32 %[sc], %[dl]\n" \
      "s_cbranch_scc1 LO0a_%=\n" \
      "s_mov_b32 %[sto], 1\n" \
      "LO0c_%=:\n" \
      "s_mov_b32 %[so0], %[sf]\n" \
      "s_mov_b64 vcc, -1\n" \
      "LSP0_%=:\n" \
      "s_mov_b64 exec, vcc\n" \
      "buffer_load_dwordx4 v[168:171], %[vs0], %[rs], %[so0] offen offset:0 sc0 sc1\n" \
      "buffer_load_dwordx4 v[172:175], %[vs0], %[rs], %[so0] offen offset:1024 sc0 sc1\n" \
      "buffer_load_dwordx4 v[176:179], %[vs1], %[rs], %[so0] offen offset:0 sc0 sc1\n" \
      "buffer_load_dwordx4 v[180:183], %[vs1], %[rs], %[so0] offen offset:1024 sc0 sc1\n" \
      "s_mov_b64 exec, -1\n" \
      ".if %[npl]\n" \
      "s_cmp_lt_u32 %[so1], 0x40000000\n" \
      "s_cbranch_scc1 LSQ0_%=\n" \
      "s_movk_i32 %[sc], %[npl]\n" \
      "LSR0_%=:\n" \
      "ds_read_b32 v238, v237 offset:576\n" \
      "s_waitcnt lgkmcnt(0)\n" \
      "s_sub_u32 %[sc], %[sc], 1\n" \
      "v_readfirstlane_b32 %[so1], v238\n" \
      "s_cbranch_scc1 LSQ0_%=\n" \
      "s_cmp_lt_u32 %[so1], 0x40000000\n" \
      "s_cbranch_scc0 LSR0_%=\n" \
      "s_nop 3\n" \
      "buffer_load_dwordx4 v[184:187], %[vs0], %[rs], %[so1] offen offset:0 sc0 sc1\n" \
      "buffer_load_dwordx4 v[188:191], %[vs0], %[rs], %[so1] offen offset:1024 sc0 sc1\n" \
      "buffer_load_dwordx4 v[192:195], %[vs1], %[rs], %[so1] offen offset:0 sc0 sc1\n" \
      "buffer_load_dwordx4 v[196:199], %[vs1], %[rs], %[so1] offen offset:1024 sc0 sc1\n" \
      "s_waitcnt vmcnt(4)\n" \
      "ds_read2_b32 v[234:235], v237 offset0:0 offset1:142\n" \
      "v_min3_u32 v232, v168, v172, v176\n" \
      "v_min_u32 v232, v232, v180\n" \
      "v_cmp_eq_u32 vcc, %[xz], v232\n" \
      "s_waitcnt lgkmcnt(0)\n" \
      "s_cbranch_vccz LS0_%=\n" \
      "LSQ0_%=:\n" \
      ".endif\n" \
      "s_memrealtime vcc\n" \
      "s_waitcnt vmcnt(0) lgkmcnt(0)\n" \
      "s_sub_u32 %[sc], vcc_lo, %[t0]\n" \
      "v_min3_u32 v232, v168, v172, v176\n" \
      "v_min_u32 v232, v232, v180\n" \
      "v_cmp_eq_u32 vcc, %[xz], v232\n" \
      "s_cbranch_vccz LSB0_%=\n" \
      "s_sleep %[psl]\n" \
      "s_cmp_lt_u32 %[sc], %[dl]\n" \
      "s_cbranch_scc1 LSP0_%=\n" \
      "s_mov_b32 %[sto], 1\n" \
      "LSB0_%=:\n" \
      "ds_read_b32 v238, v237 offset:576\n" \
      "ds_read2_b32 v[234:235], v237 offset0:0 offset1:142\n" \
      "v_min3_u32 v232, v184, v188, v192\n" \
      "v_min_u32 v232, v232, v196\n" \
      "v_cmp_eq_u32 vcc, %[xz], v232\n" \
      "s_waitcnt lgkmcnt(0)\n" \
      "v_readfirstlane_b32 %[so1], v238\n" \
      "s_or_b64 exec, vcc, 1\n" \
      "s_nop 3\n" \
      "buffer_load_dwordx4 v[184:187], %[vs0], %[rs], %[so1] offen offset:0 sc0 sc1\n" \
      "buffer_load_dwordx4 v[188:191], %[vs0], %[rs], %[so1] offen offset:1024 sc0 sc1\n" \
      "buffer_load_dwordx4 v[192:195], %[vs1], %[rs], %[so1] offen offset:0 sc0 sc1\n" \
      "buffer_load_dwordx4 v[196:199], %[vs1], %[rs], %[so1] offen offset:1024 sc0 sc1\n" \
      "s_mov_b64 exec, -1\n" \
      "s_branch LS0_%=\n" \
      "LT_%=:\n" \
      "ds_read_b32 v234, %[vsib]\n" \
      "s_sub_u32 %[sj], %[sj], 6\n" \
      "v_mfma_f32_16x16x128_f8f6f4 v[76:79], v[112:119], v[224:231], v[76:79] cbsz:1\n" \
      "v_mfma_f32_16x16x128_f8f6f4 v[80:83], v[120:127], v[224:231], v[80:83] cbsz:1\n" \
      "v_mfma_f32_16x16x128_f8f6f4 v[84:87], v[128:135], v[224:231], v[84:87] cbsz:1\n" \
      "s_waitcnt lgkmcnt(0)\n" \
      "v_mfma_f32_16x16x128_f8f6f4 v[88:91], v[136:143], v[224:231], v[88:91] cbsz:1\n" \
      "v_mfma_f32_16x16x128_f8f6f4 v[92:95], v[144:151], v[224:231], v[92:95] cbsz:1\n" \
      "v_readfirstlane_b32 %[sf], v234\n" \
      "s_cmp_gt_u32 %[sf], %[sj]\n" \
      "v_mfma_f32_16x16x128_f8f6f4 v[96:99], v[152:159], v[224:231], v[96:99] cbsz:1\n" \
      "v_mfma_f32_16x16x128_f8f6f4 v[100:103], v[160:167], v[224:231], v[100:103] cbsz:1\n" \
      "s_cbranch_scc1 LXc_%=\n" \
      "LXa_%=:\n" \
      "s_movk_i32 %[sc], 64\n" \
      "LXb_%=:\n" \
      "ds_read_b32 v234, %[vsib]\n" \
      "s_waitcnt lgkmcnt(0)\n" \
      "s_sub_u32 %[sc], %[sc], 1\n" \
      "v_readfirstlane_b32 %[sf], v234\n" \
      "s_cmp_gt_u32 %[sf], %[sj]\n" \
      "s_cbranch_scc1 LXc_%=\n" \
      "s_cmp_lg_u32 %[sc], 0\n" \
      "s_cbranch_scc1 LXb_%=\n" \
      "s_memrealtime vcc\n" \
      "s_waitcnt lgkmcnt(0)\n" \
      "s_sub_u32 %[sc], vcc_lo, %[t0]\n" \
      "s_cmp_lt_u32 %[sc], %[dl]\n" \
      "s_cbranch_scc1 LXa_%=\n" \
      "s_mov_b32 %[sto], 1\n" \
      "LXc_%=:\n" \
      "v_mfma_f32_16x16x128_f8f6f4 v[44:47], v[112:119], v[216:223], v[44:47] cbsz:1\n" \
      "ds_write_b128 %[vxo], v[72:75] offset:0\n" \
      "v_mfma_f32_16x16x128_f8f6f4 v[48:51], v[120:127], v[216:223], v[48:51] cbsz:1\n" \
      "ds_write_b128 %[vxo], v[76:79] offset:1024\n" \
      "v_mfma_f32_16x16x128_f8f6f4 v[52:55], v[128:135], v[216:223], v[52:55] cbsz:1\n" \
      "ds_write_b128 %[vxo], v[80:83] offset:2048\n" \
      "v_mfma_f32_16x16x128_f8f6f4 v[56:59], v[136:143], v[216:223], v[56:59] cbsz:1\n" \
      "ds_write_b128 %[vxo], v[84:87] offset:3072\n" \
      "v_mfma_f32_16x16x128_f8f6f4 v[60:63], v[144:151], v[216:223], v[60:63] cbsz:1\n" \
      "ds_write_b128 %[vxo], v[88:91] offset:4096\n" \
      "v_mfma_f32_16x16x128_f8f6f4 v[64:67], v[152:159], v[216:223], v[64:67] cbsz:1\n" \
      "ds_write_b128 %[vxo], v[92:95] offset:5120\n" \
      "v_mfma_f32_16x16x128_f8f6f4 v[68:71], v[160:167], v[216:223], v[68:71] cbsz:1\n" \
      "ds_write_b128 %[vxo], v[96:99] offset:6144\n" \
      "v_mov_b32 v232, %[sto]\n" \
      "ds_write_b128 %[vxo], v[100:103] offset:7168\n" \
      "v_add_u32 v232, 1, v232\n" \
      "ds_write_b32 %[vxf], v232\n" \
      "s_waitcnt vmcnt(0)\n" \
      "s_nop 7\n" \
      "s_nop 7\n"
#define MAINLOOP_CLOBBERS "v24", "v25", "v26", "v27", "v28", "v29", "v30", "v31", "v32", "v33", "v34", "v35", "v36", "v37", "v38", "v39", "v72", "v73", "v74", "v75", "v76", "v77", "v78", "v79", "v80", "v81", "v82", "v83", "v84", "v85", "v86", "v87", "v88", "v89", "v90", "v91", "v92", "v93", "v94", "v95", "v96", "v97", "v98", "v99", "v100", "v101", "v102", "v103", "v104", "v105", "v106", "v107", "v108", "v109", "v110", "v111", "v112", "v113", "v114", "v115", "v116", "v117", "v118", "v119", "v120", "v121", "v122", "v123", "v124", "v125", "v126", "v127", "v128", "v129", "v130", "v131", "v132", "v133", "v134", "v135", "v136", "v137", "v138", "v139", "v140", "v141", "v142", "v143", "v144", "v145", "v146", "v147", "v148", "v149", "v150", "v151", "v152", "v153", "v154", "v155", "v156", "v157", "v158", "v159", "v160", "v161", "v162", "v163", "v164", "v165", "v166", "v167", "v168", "v169", "v170", "v171", "v172", "v173", "v174", "v175", "v176", "v177", "v178", "v179", "v180", "v181", "v182", "v183", "v184", "v185", "v186", "v187", "v188", "v189", "v190", "v191", "v192", "v193", "v194", "v195", "v196", "v197", "v198", "v199", "v200", "v201", "v202", "v203", "v204", "v205", "v206", "v207", "v208", "v209", "v210", "v211", "v212", "v213", "v214", "v215", "v216", "v217", "v218", "v219", "v220", "v221", "v222", "v223", "v224", "v225", "v226", "v227", "v228", "v229", "v230", "v231", "v232", "v233", "v234", "v235", "v236", "v237", "v238"

__global__ __launch_bounds__(NWAVES * 64, 1) void a2a_gemm_kernel(Params P) {
  __shared__ __attribute__((aligned(16))) char lds[LDS_BYTES];
  asm volatile("s_load_dwordx16 s[64:79], %0, 0x0\n\ts_load_dwordx16 s[80:95], %0, 0x40\n\ts_memrealtime s[98:99]"
               ::"s"(__builtin_amdgcn_kernarg_segment_ptr())
               : "s64", "s65", "s66", "s67", "s68", "s69", "s70", "s71", "s72", "s73", "s74", "s75", "s76", "s77", "s78", "s79",
                 "s80", "s81", "s82", "s83", "s84", "s85", "s86", "s87", "s88", "s89", "s90", "s91", "s92", "s93", "s94", "s95",
                 "s98", "s99");
#define STAMP(i)
  const int tid = threadIdx.x;
  const int w = __builtin_amdgcn_readfirstlane(tid >> 6);
  const int l = tid & 63;
  const int b = blockIdx.x & 255;
  const int xcd = b & 7, idx = b >> 3;
  const int mb = idx & 7;
#define NBMAP 0
  const int nb = NBMAP == 1 ? xcd * 4 + 3 - (idx >> 3) : NBMAP == 2 ? (xcd ^ 1) * 4 + (idx >> 3) : NBMAP == 3 ? xcd + 8 * (idx >> 3) : xcd * 4 + (idx >> 3);
  const int nbar = ((b & (256 / PG - 1)) + 256 - NE) >> 8;
  const unsigned ldsb = (unsigned)(size_t)lds;
  volatile LDSP unsigned* full = (volatile LDSP unsigned*)(lds + CTRL_OFF);
  volatile LDSP v2u* done = (volatile LDSP v2u*)(lds + DONE_OFF);

#define ARGS \
    v4i rw; unsigned long long xb; char* myws; float* outp; const float* psx; const float* psw; int rank; unsigned t0; \
    asm volatile("s_waitcnt lgkmcnt(0)" : "={s[64:67]}"(rw), "={s[68:69]}"(xb), "={s[70:71]}"(myws), "={s[72:73]}"(outp), \
                 "={s[74:75]}"(psx), "={s[76:77]}"(psw), "={s78}"(rank), "={s98}"(t0) \
                 : "s"(__builtin_amdgcn_kernarg_segment_ptr()) : "memory"); \
    rank &= 7; \
    GLBP unsigned* ctr = (GLBP unsigned*)(myws + b * 64 + w * 4); \
    const v4i rx = {(int)(unsigned)xb, (int)(unsigned)(xb >> 32), 4096 * 512, 0x00020000}; \
    (void)ctr; (void)rx;

  if (w == 1) {
    full[l] = 0;
    ((volatile LDSP unsigned*)(lds + FLG_OFF))[l] = 0;
    ((volatile LDSP unsigned*)(lds + ORD_OFF))[l] = OOR;
    ((volatile LDSP unsigned*)(lds + ORW_OFF))[l] = OOR;
    const unsigned f0 = (unsigned)((l >> 4) & 3);
    const unsigned v0 = (unsigned)(l >> 3) * 4096u + ((((unsigned)l & 7u) ^ f0) << 4) + nb * 128;
    const unsigned pl = (unsigned)(l >> 2) * 4096u + (nb >> 2) * 512 + (l & 3) * 128;
    unsigned pown = pl + idx * 16 * 4096u;
    unsigned pslb = pl + ((idx >> 2) * 512 + (idx & 3) * 32) * 4096u;
    unsigned vda = ldsb + DONE_OFF, vor = ldsb + ORW_OFF + 47 * 4, mf = ldsb + FLG_OFF + 124;
    unsigned a0 = v0, a1 = v0 + 31744u, a2 = (v0 ^ 64u) + 2 * 31744u, a3 = (v0 ^ 64u) + 3 * 31744u;
    unsigned a4 = a0 + 131072u, a5 = a1 + 131072u, a6 = a2 + 131072u, a7 = a3 + 131072u, vz = 0;
    const int li = (l - 4) & 31, ls = li / 7, lu = li - ls * 7, lt = lu + (lu >= mb ? 1 : 0);
    unsigned vord = (unsigned)(((lt - mb) & 7) << 18 | (ls & 3) << 16);
    asm volatile("" : "+v"(a0), "+v"(a1), "+v"(a2), "+v"(a3), "+v"(a4), "+v"(a5), "+v"(a6), "+v"(a7), "+v"(vz),
                 "+v"(pown), "+v"(pslb), "+v"(vda), "+v"(vor), "+v"(vord));
    ARGS
    unsigned md = ldsb, jn = 0, so = (unsigned)rank << 21, sd, sq;
    unsigned long long pc;
    v2u vd;
#define WDMA(i, o, s) "buffer_load_dwordx4 %[a" #i "], %[rw], %[" #s "] offen offset:" #o " lds\n"
#define WPF(a, s) "buffer_load_dword v22, %[" #a "], %[rw], " s " offen\n"
    asm volatile(
        "s_mov_b32 %[sq], 0x10000\n"
        "s_cmp_lg_u32 %[nbar], 0\n"
        "s_cbranch_scc1 LA_%=\n"
        "s_barrier\n"
        "LA_%=:\n"
        ".if %[vpi] & 1\n"
        WPF(pown, "%[so]")
        ".endif\n"
        ".if %[vpi] & 2\n"
        WPF(pslb, "0") WPF(pslb, "%[sq]")
        ".endif\n"
        "s_getpc_b64 %[pc]\n"
        "LI_%=:\n"
        "s_mov_b32 m0, %[md]\n"
        "s_add_u32 %[sd], %[so], 0x40000\n"
        WDMA(0, 0, so) WDMA(1, 1024, so) WDMA(2, 2048, so) WDMA(3, 3072, so)
        "s_add_u32 m0, %[md], 0x1000\n"
        "s_add_u32 %[jn], %[jn], 1\n"
        WDMA(4, 0, so) WDMA(5, 1024, so) WDMA(6, 2048, so) WDMA(7, 3072, so)
        "s_add_u32 m0, %[md], 0x2000\n"
        "s_add_u32 %[so], %[so], 0x80000\n"
        WDMA(0, 0, sd) WDMA(1, 1024, sd) WDMA(2, 2048, sd) WDMA(3, 3072, sd)
        "s_add_u32 m0, %[md], 0x3000\n"
        "s_add_u32 %[md], %[md], 0x4000\n"
        WDMA(4, 0, sd) WDMA(5, 1024, sd) WDMA(6, 2048, sd) WDMA(7, 3072, sd)
        "s_mov_b32 m0, %[mf]\n"
        "s_sub_u32 %[mf], %[mf], 4\n"
        "global_load_lds_dword %[vz], %[pc]\n"
        "s_and_b32 %[md], %[md], 0x1ffff\n"
        "v_xor_b32 %[vda], 8, %[vda]\n"
        "v_add_u32 %[vor], -4, %[vor]\n"
        "s_cmp_ge_u32 %[jn], 32\n"
        "s_cbranch_scc1 LE_%=\n"
        "ds_read_b64 %[vd], %[vda]\n"
        "ds_read_b32 v23, %[vor]\n"
        "s_cmp_lg_u32 %[jn], %[nbar]\n"
        "s_cbranch_scc1 LB_%=\n"
        "s_barrier\n"
        "LB_%=:\n"
        "s_waitcnt vmcnt(%[lde])\n"
        "s_cmp_lt_u32 %[jn], %[ldn]\n"
        "s_cbranch_scc1 LD_%=\n"
        "s_waitcnt vmcnt(%[ldw])\n"
        "LD_%=:\n"
        ".if %[vpf]\n"
        "s_cmp_lg_u32 %[jn], %[vpf]\n"
        "s_cbranch_scc1 LN_%=\n"
        "s_mov_b32 %[sq], 0x80000\n"
        "s_mov_b32 %[sd], 0x90000\n"
        WPF(pslb, "%[sq]") WPF(pslb, "%[sd]")
        "LN_%=:\n"
        ".endif\n"
        "s_cmp_lt_u32 %[jn], 4\n"
        "s_cbranch_scc1 LI_%=\n"
        "LQ_%=:\n"
        "s_movk_i32 %[sq], 64\n"
        "LR_%=:\n"
        "s_waitcnt lgkmcnt(0)\n"
        "v_min_u32 v20, v20, v21\n"
        ".if %[stat]\n"
        "v_readlane_b32 %[so], %[vord], %[jn]\n"
        ".else\n"
        "v_readfirstlane_b32 %[so], v23\n"
        ".endif\n"
        "v_readfirstlane_b32 %[sd], v20\n"
        "s_sub_u32 %[sq], %[sq], 1\n"
        "s_add_u32 %[sd], %[sd], 7\n"
        "s_cmp_ge_u32 %[sd], %[jn]\n"
        "s_cselect_b32 %[sd], %[so], -1\n"
        ".if %[stat]\n"
        "s_add_u32 %[so], %[so], %[rk]\n"
        ".endif\n"
        "s_and_b32 %[so], %[so], 0x1fffff\n"
        "s_lshl_b32 %[so], %[so], 3\n"
        "s_cmp_lt_u32 %[sd], 0x40000000\n"
        "s_cbranch_scc1 LI_%=\n"
        "ds_read_b64 %[vd], %[vda]\n"
        "ds_read_b32 v23, %[vor]\n"
        "s_cmp_lg_u32 %[sq], 0\n"
        "s_cbranch_scc1 LR_%=\n"
        "s_memrealtime vcc\n"
        "s_waitcnt lgkmcnt(0)\n"
        "s_sub_u32 %[sq], vcc_lo, %[t0]\n"
        "s_cmp_lt_u32 %[sq], %[dl]\n"
        "s_cbranch_scc1 LQ_%=\n"
        "s_branch LI_%=\n"
        "LE_%=:\n"
        "s_waitcnt vmcnt(0) lgkmcnt(0)\n"
        : [vda] "+v"(vda), [vor] "+v"(vor), [md] "+&s"(md), [mf] "+&s"(mf), [jn] "+&s"(jn), [so] "+&s"(so),
          [sd] "=&s"(sd), [sq] "=&s"(sq), [pc] "=&s"(pc), [vd] "=&{v[20:21]}"(vd)
        : [a0] "v"(a0), [a1] "v"(a1), [a2] "v"(a2), [a3] "v"(a3), [a4] "v"(a4), [a5] "v"(a5), [a6] "v"(a6), [a7] "v"(a7),
          [rw] "s"(rw), [t0] "s"(t0), [vz] "v"(vz),
          [nbar] "s"(nbar), [pown] "v"(pown), [pslb] "v"(pslb), [ldw] "i"(LDW), [vpf] "i"(VPF), [vpi] "i"(VPI), [dl] "i"(2 * DEADLINE),
          [stat] "i"(STAT), [lde] "i"(LDE), [ldn] "i"(LDN), [vord] "v"(vord), [rk] "s"(rank << 18)
        : "vcc", "scc", "memory", "v22", "v23");
    __builtin_amdgcn_endpgm();
  }
  if (w == 0) {
    __builtin_amdgcn_s_setprio(PPRIO);
    const int pb = (idx < RV ? idx : 31 + RV - idx) * 8 + xcd;
    int s = pb >> 6, tau = (pb >> 3) & 7, e = pb & 7;
    const int row = (e & 1) * 8 + (l >> 3), ch = l & 7;
    unsigned xoff = row * 512 + ch * 16 + (e >> 1) * 8192 + s * 128, coff = b * 64;
    unsigned h0off = WS_HELLO + (b & 31) * 128 + (l & 7) * 4;
    asm volatile("" : "+v"(xoff), "+v"(coff), "+v"(h0off));
    ARGS
    char* peer[8];
    asm volatile("" : "={s[82:83]}"(peer[1]), "={s[84:85]}"(peer[2]), "={s[86:87]}"(peer[3]),
                 "={s[88:89]}"(peer[4]), "={s[90:91]}"(peer[5]), "={s[92:93]}"(peer[6]), "={s[94:95]}"(peer[7]));
    unsigned cv;
    asm volatile("global_load_dword %0, %1, %2 sc0 sc1" : "=v"(cv) : "v"(coff), "s"(myws) : "memory");
    unsigned h0 = 0;
    if (PS) asm volatile("global_load_dword %0, %1, %2 sc0 sc1" : "=v"(h0) : "v"(h0off), "s"(myws) : "memory");
    v4u v[8];
#pragma unroll
    for (int d = 1; d < 8; ++d) {
      const unsigned so = (((rank + d) & 7) << 18) | (((tau + d) & 7) << 15);
      asm volatile("buffer_load_dwordx4 %0, %1, %2, %3 offen" : "=v"(v[d]) : "v"(xoff), "s"(rx), "s"(so) : "memory");
    }
    STAMP(0);
    __builtin_amdgcn_s_barrier();
    STAMP(1);
    asm volatile("" : "+s"(s), "+s"(tau), "+s"(e));
    const unsigned doff = s * 65536 + (e >> 1) * 2048 + (ch >> 2) * 1024 + ((ch & 3) * 16 + row) * 16;
    unsigned tgo = PACE0 + (unsigned)(pb * (PACE - 7 * PS)) / 100u;
    const int ks = l / 7, ku = l - ks * 7, kt = ku + (ku >= mb ? 1 : 0), kp = (rank - ((mb - kt) & 7)) & 7;
    const unsigned koff = (unsigned)((64 * ks + 8 * kt) * HP) / 100u;
    unsigned key = STAT && l < 28 ? HK + koff : ~0u;
    const unsigned hoff = WS_HELLO + (b & 31) * 128 + kp * 4;
    unsigned hv = 0;
    const bool look = !STAT && tgo > HT + HQ;
    if (look) {
      while (now() - t0 < HT) __builtin_amdgcn_s_sleep(1);
      asm volatile("global_load_dword %0, %1, %2 sc0 sc1" : "=v"(hv) : "v"(hoff), "s"(myws) : "memory");
    }
    asm volatile("s_waitcnt vmcnt(0)" : "+v"(cv), "+v"(hv), "+v"(h0), "+v"(v[1]), "+v"(v[2]), "+v"(v[3]), "+v"(v[4]), "+v"(v[5]), "+v"(v[6]), "+v"(v[7]) :: "memory");
    const unsigned c = __builtin_amdgcn_readfirstlane(cv);
    const unsigned so = (c % 3u) * STAGE_BYTES;
    const unsigned desc = so + kp * 262144 + ks * 65536;
    if (PS) tgo = PACE0 + (unsigned)(pb * (PACE - PS * __builtin_popcount((unsigned)__builtin_amdgcn_ballot_w64((int)(h0 - c) > 0) & 0xffu))) / 100u;
#pragma unroll
    for (int d = 1; d < 8; ++d) v[d][0] = v[d][0] ? v[d][0] : 0x80u;
    STAMP(2);
#define ZMOD " sc0 sc1"
#define clean() { \
      if (l == 0) asm volatile("global_store_dword %0, %1, %2" ZMOD ::"v"(coff), "v"(c + 1u), "s"(myws) : "memory"); \
      const unsigned z = WS_STAGE + ((c + 2u) % 3u) * STAGE_BYTES + b * 1024 + l * 16; \
      const v4u zero = {0u, 0u, 0u, 0u}; \
      _Pragma("unroll") \
      for (int i = 1; i < 8; ++i) { \
        const unsigned zo = z + ((rank + i) & 7) * 262144; \
        asm volatile("global_store_dwordx4 %0, %1, %2" ZMOD "\n\ts_nop 1" ::"v"(zo), "v"(zero), "s"(myws) : "memory"); \
      } }
    if (ZT == 1 && b >= NE) clean();
    bool pushed = false;
    unsigned open = 0xfffffffu, unseen = 0xfffffffu, jc = ldsb + ORD_OFF + 43 * 4, vt = 0, pos = STAT ? jc - 4 * l : 0u, hd = 0;
    if (look) {
      if (l < 28 && (int)(hv - c) > 0) key = HK + koff;
      unseen = (unsigned)__builtin_amdgcn_ballot_w64(key == ~0u) & 0xfffffffu;
      hd = HT + HE;
    }
    while (true) {
      unsigned t = now() - t0;
      if (!pushed && t >= tgo) {
        STAMP(5);
        pushed = true;
#pragma unroll
        for (int d = 1; d < 8; ++d) {
          const unsigned off = doff + so + ((tau + d) & 7) * 8192;
          if (!NOPUSH) asm volatile("global_store_dwordx4 %0, %1, %2 sc0 sc1\n\ts_nop 1" ::"v"(off), "v"(v[d]), "s"(peer[d]) : "memory");
        }
        STAMP(6);
        if (ZT == 0 || (ZT == 1 && b < NE)) clean();
      }
      if (!STAT && open) {
        vt = unseen ? 0u : vt + HS;
        vt = vt > t + HX ? vt : t + HX;
        const unsigned m = (unsigned)__builtin_amdgcn_ballot_w64(key <= vt) & open;
        if (m) {
          if (m >> (l & 31) & (l < 32 ? 1 : 0)) {
            pos = jc - 4 * __builtin_amdgcn_mbcnt_lo(m, 0u);
            *(volatile LDSP unsigned*)(size_t)(pos + (ORW_OFF - ORD_OFF)) = desc;
          }
          jc -= 4 * __builtin_popcount(m);
          open &= ~m;
        }
      }
      if (key <= t) { *(volatile LDSP unsigned*)(size_t)pos = desc; key = ~1u; }
      if (!STAT && unseen && (pushed || t + HQ < tgo)) {
        asm volatile("global_load_dword %0, %1, %2 sc0 sc1\n\ts_waitcnt vmcnt(0)" : "=v"(hv) : "v"(hoff), "s"(myws) : "memory");
        if (l < 28 && key == ~0u && ((int)(hv - c) > 0 || t > 5000u)) key = (hd > HK ? hd : HK) + (hd > HL ? koff * HPF / HP : koff);
        unseen = (unsigned)__builtin_amdgcn_ballot_w64(key == ~0u) & 0xfffffffu;
        hd = t + HE;
        continue;
      }
      if (pushed && ((unsigned)__builtin_amdgcn_ballot_w64(key != ~1u) & 0xfffffffu) == 0u) break;
      __builtin_amdgcn_s_sleep(1);
    }
    if (ZT >= 2) {
      while (ZT > 2 && ((volatile LDSP unsigned*)(lds + FLG_OFF))[31 - ZT] == 0u && now() - t0 < DEADLINE) __builtin_amdgcn_s_sleep(4);
      clean();
    }
    __builtin_amdgcn_endpgm();
  }
  if (w == 2 || (!NMON && w == 7)) {
    ARGS
    __builtin_amdgcn_s_barrier();
    if (!STAT && w == 2 && b < 8 && l < 4) {
      const unsigned ho = WS_HELLO - WS_STAGE - rank * 262140 + (b * 4 + l) * 128;
#define HEL(r) "global_atomic_add %0, %1, " r " sc1\n\t"
      asm volatile(HEL("s[82:83]") HEL("s[84:85]") HEL("s[86:87]") HEL("s[88:89]") HEL("s[90:91]") HEL("s[92:93]") HEL("s[94:95]")
                   :: "v"(ho), "v"(1u) : "memory");
    }
    const char* wp = (const char*)((unsigned long long)(unsigned)rw[0] | (unsigned long long)(unsigned)rw[1] << 32) + (size_t)(NMON ? 384 + mb * 16 : SP0 * 128 + mb * 16 + (w & 1) * 8) * 4096 + nb * 128 + 60;
    unsigned i = 1, sp, st;
#define SPL(k) "s_load_dwordx2 s[96:97], %[wp], %[sp] offset:" #k "*4096\n"
    asm volatile(
        "LP_%=:\n"
        "s_add_u32 %[sp], %[i], %[rk]\n"
        "s_and_b32 %[sp], %[sp], 7\n"
        "s_lshl_b32 %[sp], %[sp], 21\n"
        "s_and_b32 %[st], %[i], 24\n"
        "s_lshl_b32 %[st], %[st], %[sh]\n"
        "s_add_u32 %[sp], %[sp], %[st]\n"
        SPL(0) SPL(1) SPL(2) SPL(3) SPL(4) SPL(5) SPL(6) SPL(7)
        "s_add_u32 %[i], %[i], 1\n"
        "s_and_b32 %[st], %[i], 7\n"
        "s_cmp_eq_u32 %[st], 0\n"
        "s_addc_u32 %[i], %[i], 0\n"
        "s_cmp_lt_u32 %[i], %[ie]\n"
        "s_cbranch_scc1 LP_%=\n"
        "s_waitcnt lgkmcnt(0)\n"
        : [i] "+s"(i), [sp] "=&s"(sp), [st] "=&s"(st)
        : [wp] "s"(wp), [rk] "s"(rank), [sh] "i"(NMON ? 12 : 16), [ie] "i"(NMON ? 16 : 32 - 8 * SP0)
        : "scc", "memory", "s96", "s97");
    __builtin_amdgcn_endpgm();
  }
  if (w < 1 + NL + 4) {
    const int mw = w - 1 - NL;
    const int par = mw >> 1, hw = mw & 1;
    const int g = l >> 4, r = l & 15;
    const int rb = mb * 4 + hw * 2 + par;
    unsigned vs0 = rb * 2048 + l * 16, vs1 = vs0 ^ 2048u;
    unsigned vx0 = (rb * 16 + r) * 512 + g * 16 + par * 128, vx1 = vx0 ^ 8192u;
    const int kk = 16 * g + (r >> 1);
    const int fsw = ((kk >> 1) & 3) | (((kk >> 4) & 1) << 2);
    unsigned ab = ldsb + par * 16384 + kk * 128 + (l & 1) * 8 + fsw * 16;
    unsigned vdone = ldsb + DONE_OFF + (l * 4 + mw) * 4;
    const unsigned ctl = ldsb + FLG_OFF + 124 - par * 4 - 8;
    unsigned vsib = ldsb + DONE_OFF + (mw ^ 1) * 4;
    unsigned vxo = ldsb + par * 16384 + hw * 8192 + l * 16;
    unsigned vxf = ldsb + CTRL_OFF + 64 + mw * 4;
    asm volatile("" : "+v"(vs0), "+v"(vs1), "+v"(vx0), "+v"(vx1), "+v"(ab), "+v"(vdone), "+v"(vsib), "+v"(vxo), "+v"(vxf));
    ARGS
    const unsigned long long sb = (unsigned long long)(myws + WS_STAGE);
    const v4i rs = {(int)(unsigned)sb, (int)(unsigned)(sb >> 32), (int)(3 * STAGE_BYTES), 0x00020000};
    v16f c0, c1;
    float ssx, ssw;
    unsigned sto, sj, sf, so0, so1, so2, so3, sn, sc;
    asm volatile(MAINLOOP_ASM
                 : "=&{v[40:55]}"(c0), "=&{v[56:71]}"(c1),
                   [ssx] "=&s"(ssx), [ssw] "=&s"(ssw), [sto] "=&s"(sto),
                   [sj] "=&s"(sj), [sf] "=&s"(sf), [so0] "=&s"(so0), [so1] "=&s"(so1), [so2] "=&s"(so2), [so3] "=&s"(so3),
                   [sn] "=&s"(sn), [sc] "=&s"(sc)
                 : [ab] "v"(ab), [vs0] "v"(vs0), [vs1] "v"(vs1), [vdone] "v"(vdone), [rs] "s"(rs),
                   [psx] "s"(psx), [psw] "s"(psw), [xo] "s"(rank << 18), [t0] "s"(t0), [ctl] "s"(ctl), [dl] "i"(DEADLINE),
                   [vx0] "v"(vx0), [vx1] "v"(vx1), [rx] "s"(rx), [psl] "i"(POLLSLEEP), [par] "s"(par),
                   [vsib] "v"(vsib), [vxo] "v"(vxo), [vxf] "v"(vxf), [x1] "i"(X1), [nbar] "s"(nbar), [xz] "i"(XFAKE ? -1 : 0), [xsc] "i"(XSC), [npl] "i"(NPL)
                 : MAINLOOP_CLOBBERS, "vcc", "scc", "memory");
    unsigned pf;
    while ((pf = (unsigned)__builtin_amdgcn_readfirstlane(full[16 + (mw ^ 2)])) == 0u) { if (now() - t0 > 2 * DEADLINE) { pf = 2; break; } }
    volatile LDSP v4f* xi = (volatile LDSP v4f*)(lds + (par ^ 1) * 16384 + hw * 8192) + l;
    v4f pa[8];
#pragma unroll
    for (int t = 0; t < 8; ++t) pa[t] = xi[t * 64];
    const float scale = (sto || pf != 1u) ? __builtin_nanf("") : ssx * ssw;
    const float sl = scale * -1.4426950408889634f, lim = 60.f * 1.4426950408889634f;
    float* orow = outp + (size_t)(rb * 16 + r) * 4096 + nb * 128 + g * 4;
#pragma unroll
    for (int t = 0; t < 8; ++t) {
      v4f o;
#pragma unroll
      for (int e = 0; e < 4; e += 2) {
        v2f a = {t < 4 ? c0[(t & 3) * 4 + e] : c1[(t & 3) * 4 + e], t < 4 ? c0[(t & 3) * 4 + e + 1] : c1[(t & 3) * 4 + e + 1]};
        a += (v2f){pa[t][e], pa[t][e + 1]};
        const v2f y = a * scale, z = a * sl;
        v2f d = {__builtin_amdgcn_exp2f(__builtin_amdgcn_fmed3f(z[0], -lim, lim)), __builtin_amdgcn_exp2f(__builtin_amdgcn_fmed3f(z[1], -lim, lim))};
        d += 1.f;
        const v2f q = y * (v2f){__builtin_amdgcn_rcpf(d[0]), __builtin_amdgcn_rcpf(d[1])};
        o[e] = q[0]; o[e + 1] = q[1];
      }
      asm volatile("global_store_dwordx4 %0, %1, off sc0 sc1\n\ts_nop 1" ::"v"(orow + t * 16), "v"(o) : "memory");
    }
  } else {
  }
}

extern "C" void kernel_launch(int rank, int nranks, ncclComm_t comm, hipStream_t stream,
                              void* const* d_in, const int* in_sizes, int n_in,
                              void* d_out, int out_size, const int* in_dtypes, int out_dtype,
                              void* const* d_out_all, void* const* d_ws_all, size_t ws_bytes) {
  if (nranks != 8 || (unsigned)rank >= 8u || n_in < 4 || ws_bytes < (size_t)(WS_HELLO + 4096) ||
      out_size != 512 * 4096 || in_sizes[0] != 4096 * 512 || in_sizes[1] != 4096 * 4096) {
    fprintf(stderr, "unexpected problem shape\n");
    return;
  }
  Params P;
  const unsigned long long wb = (unsigned long long)d_in[1];
  if ((wb | (unsigned long long)d_in[0] | (unsigned long long)d_ws_all[rank]) >> 47) {
    fprintf(stderr, "unexpected address\n");
    return;
  }
  P.rw[0] = (unsigned)wb; P.rw[1] = (unsigned)(wb >> 32); P.rw[2] = 4096u * 4096u; P.rw[3] = 0x00020000u;
  P.x = (const char*)d_in[0];
  P.sx = (const float*)d_in[2]; P.sw = (const float*)d_in[3];
  P.out = (float*)d_out;
  P.myws = (char*)d_ws_all[rank];
  for (int d = 0; d < 8; ++d) P.peer[d] = (char*)d_ws_all[(rank + d) & 7] + WS_STAGE + rank * 262144;
  P.rank = rank; P.pad0 = 0;
  hipLaunchKernelGGL(a2a_gemm_kernel, dim3(256), dim3(NWAVES * 64), 0, stream, P);
}
```
